```python
import numpy as np

kernel_name = "gpu_gemm_rs_m4096_k4096_n4096_bf16_qe4m3"


ABSMAX_RTOL = 0.09

COMPUTE_DTYPE = "bfloat16"
EPILOGUE = "quant_e4m3"
IN_QUANT = None
OUT_QUANT = "e4m3"
IN_DTYPES = {"x": "bfloat16", "w_mat": "bfloat16"}
WORLD_SIZE = 8

def _to_bf16(x):
    u = np.ascontiguousarray(x, dtype=np.float32).view(np.uint32)
    return ((u + 0x7FFF + ((u >> 16) & 1)) & 0xFFFF0000).view(np.float32)

def setup_inputs():

    x = _to_bf16(np.random.randn(4096, 4096).astype(np.float32))
    w_mat = _to_bf16((np.random.randn(4096, 4096) / np.sqrt(4096)).astype(np.float32))
    return {"x": x, "w_mat": w_mat}

def _snap_fp8(x, mant_bits, max_abs, min_normal):
    a = np.abs(x.astype(np.float64))
    _m, _e = np.frexp(a)
    q = np.where(a >= min_normal, np.ldexp(1.0, _e - 1 - mant_bits),
                 min_normal * 2.0 ** -mant_bits)
    snapped = np.minimum(np.round(a / q) * q, max_abs)
    return (np.sign(x) * snapped).astype(np.float32)
def _quant_dequant(y, amax):
    scale = amax / 448.0
    q = _snap_fp8(y / scale, 3, 448.0, 2.0 ** -6)
    return (q * scale).astype(np.float32)
def _epilogue(y):
    return _quant_dequant(y, np.abs(y).max())

def reference(x, w_mat):
    y = (x @ w_mat).astype(np.float32)
    return _epilogue(y)
```

```cpp
#include <hip/hip_runtime.h>
#include <rccl/rccl.h>
#include <cstdio>
#include <cstdint>

#define LAS __attribute__((address_space(3)))
#define GAS __attribute__((address_space(1)))

namespace gr {

typedef unsigned u32;
typedef unsigned long long u64;
typedef unsigned u32x4 __attribute__((ext_vector_type(4)));
typedef unsigned short u16x2 __attribute__((ext_vector_type(2)));
typedef short s16x4 __attribute__((ext_vector_type(4)));
typedef short s16x8 __attribute__((ext_vector_type(8)));
typedef float f32x4 __attribute__((ext_vector_type(4)));

constexpr int NR = 8;
constexpr int NS = 32;
constexpr int NGEMM = 128;
constexpr int NCOMM = 128;
constexpr int NPUSHWG = 56;
constexpr int PUSH_PER_DST = 16;
constexpr int NUNP = NPUSHWG * 5 + (NCOMM - NPUSHWG) * 8;
constexpr int NPOS_IN = 512;
constexpr int NPOS_OUT = 256;
constexpr u64 DEADLINE = 40000000ull;

constexpr size_t OFF_CTL = 0;
constexpr size_t OFF_AMAX = 4096;
constexpr size_t OFF_RDY = 16384;
constexpr size_t OFF_HIN = 32768;
constexpr size_t OFF_HP = OFF_HIN + 65536;
constexpr size_t OFF_HQ = OFF_HP + 32768;
constexpr size_t OFF_DBG = 262144;
constexpr size_t OFF_IN = 1u << 20;
constexpr size_t OFF_IMGA = OFF_IN + (16u << 20);
constexpr size_t OFF_IMGB = OFF_IMGA + (8u << 20);
constexpr size_t OFF_P = OFF_IMGB + (8u << 20);
constexpr size_t OFF_Q = OFF_P + (8u << 20);
constexpr size_t WS_NEED = OFF_Q + (8u << 20);
constexpr int LDS_BYTES = 131072 + 1024;

struct Params {
    const char* x;
    const char* w;
    char* out;
    char* ws[NR];
    int rank;
    int pad;
};

#define WAITV(n) asm volatile("s_waitcnt vmcnt(" #n ")" ::: "memory")
#define BAR() __builtin_amdgcn_s_barrier()
#define SCHED() __builtin_amdgcn_sched_barrier(0)

__device__ __forceinline__ u64 rt() { return __builtin_amdgcn_s_memrealtime(); }
__device__ __forceinline__ u32 rfl(u32 v) { return (u32)__builtin_amdgcn_readfirstlane((int)v); }

#define STAMP(slot) do { } while (0)

__device__ __forceinline__ const char* uni(const char* p) {
    const u64 v = (u64)p;
    return (const char*)(((u64)rfl((u32)(v >> 32)) << 32) | (u64)rfl((u32)v));
}
template <int OFF> __device__ __forceinline__ void st_sys(const char* sbase, u32 voff, u32x4 v) {
    asm volatile("s_nop 4\n\tglobal_store_dwordx4 %0, %1, %2 offset:%3 sc0 sc1\n\ts_nop 1" :: "v"(voff), "v"(v), "s"(uni(sbase)), "i"(OFF) : "memory");
}
template <int OFF> __device__ __forceinline__ void st_agt(const char* sbase, u32 voff, u32x4 v) {
    asm volatile("s_nop 4\n\tglobal_store_dwordx4 %0, %1, %2 offset:%3 sc1\n\ts_nop 1" :: "v"(voff), "v"(v), "s"(uni(sbase)), "i"(OFF) : "memory");
}
__device__ __forceinline__ u32x4 poll16(const char* sbase) {
    u32x4 r; u32 z = 0;
    asm volatile("s_nop 4\n\tglobal_load_dwordx4 %0, %1, %2 sc0 sc1\n\ts_waitcnt vmcnt(0)" : "=&v"(r) : "v"(z), "s"(uni(sbase)) : "memory");
    return r;
}
__device__ __forceinline__ void ld3_sys(u32x4& a, u32x4& b, u32x4& c, const char* sbase, u32 voff) {
    asm volatile("s_nop 4\n\tglobal_load_dwordx4 %0, %3, %4 sc0 sc1\n\tglobal_load_dwordx4 %1, %3, %4 offset:1024 sc0 sc1\n\t"
                 "global_load_dwordx4 %2, %3, %4 offset:2048 sc0 sc1\n\ts_waitcnt vmcnt(0)"
                 : "=&v"(a), "=&v"(b), "=&v"(c) : "v"(voff), "s"(uni(sbase)) : "memory");
}
__device__ __forceinline__ void ld4_sys(u32x4& a, u32x4& b, u32x4& c, u32x4& d, const char* sbase, u32 voff) {
    asm volatile("s_nop 4\n\tglobal_load_dwordx4 %0, %4, %5 sc0 sc1\n\tglobal_load_dwordx4 %1, %4, %5 offset:1024 sc0 sc1\n\t"
                 "global_load_dwordx4 %2, %4, %5 offset:2048 sc0 sc1\n\tglobal_load_dwordx4 %3, %4, %5 offset:3072 sc0 sc1\n\ts_waitcnt vmcnt(0)"
                 : "=&v"(a), "=&v"(b), "=&v"(c), "=&v"(d) : "v"(voff), "s"(uni(sbase)) : "memory");
}
__device__ __forceinline__ void ld4_issue(u32x4 (&r)[4], const char* sbase, u32 v0, u32 v1, u32 v2, u32 v3) {
    asm volatile("s_nop 4\n\tglobal_load_dwordx4 %0, %4, %8 sc1\n\tglobal_load_dwordx4 %1, %5, %8 sc1\n\t"
                 "global_load_dwordx4 %2, %6, %8 sc1\n\tglobal_load_dwordx4 %3, %7, %8 sc1"
                 : "=&v"(r[0]), "=&v"(r[1]), "=&v"(r[2]), "=&v"(r[3]) : "v"(v0), "v"(v1), "v"(v2), "v"(v3), "s"(uni(sbase)) : "memory");
}
#define PIN4(r) asm volatile("" : "+v"(r[0]), "+v"(r[1]), "+v"(r[2]), "+v"(r[3]))

__device__ __forceinline__ u32x4 lds_poll16(u32 addr) {
    u32x4 r;
    asm volatile("ds_read_b128 %0, %1\n\ts_waitcnt lgkmcnt(0)" : "=&v"(r) : "v"(addr) : "memory");
    return r;
}
struct Ctx { const char* tmo; u32 call; bool dead; };
__device__ __forceinline__ bool give_up(Ctx& c, u64 t0, u32& polls) {
    __builtin_amdgcn_s_sleep(2);
    ++polls;
    if ((polls & 15u) == 0u) {
        u32 t = __hip_atomic_load((const GAS u32*)c.tmo, __ATOMIC_RELAXED, __HIP_MEMORY_SCOPE_AGENT);
        if (rfl(t) == c.call) { c.dead = true; return true; }
        if (rt() - t0 > DEADLINE) {
            __hip_atomic_store((GAS u32*)c.tmo, c.call, __ATOMIC_RELAXED, __HIP_MEMORY_SCOPE_AGENT);
            c.dead = true; return true;
        }
    }
    return false;
}
__device__ __forceinline__ bool wait_hdr(Ctx& c, const char* hp, u32& a, u32& b) {
    a = 0; b = 0;
    if (c.dead) return false;
    const u64 t0 = rt(); u32 polls = 0;
    for (;;) {
        u32x4 h = poll16(hp);
        const u32 s0 = rfl(h.x), s1 = rfl(h.z);
        if (s0 >= c.call && s1 >= c.call) { a = rfl(h.y); b = rfl(h.w); return true; }
        if (give_up(c, t0, polls)) return false;
    }
}

__device__ __forceinline__ u32 perm(u32 s0, u32 s1, u32 sel) { return __builtin_amdgcn_perm(s0, s1, sel); }
__device__ __forceinline__ u32 wave_umax(u32 v) {
    v = max(v, (u32)__builtin_amdgcn_update_dpp(0, (int)v, 0x111, 0xf, 0xf, true));
    v = max(v, (u32)__builtin_amdgcn_update_dpp(0, (int)v, 0x112, 0xf, 0xf, true));
    v = max(v, (u32)__builtin_amdgcn_update_dpp(0, (int)v, 0x114, 0xf, 0xf, true));
    v = max(v, (u32)__builtin_amdgcn_update_dpp(0, (int)v, 0x118, 0xf, 0xf, true));
    v = max(v, (u32)__builtin_amdgcn_update_dpp(0, (int)v, 0x142, 0xa, 0xf, true));
    v = max(v, (u32)__builtin_amdgcn_update_dpp(0, (int)v, 0x143, 0xc, 0xf, true));
    return (u32)__builtin_amdgcn_readlane((int)v, 63);
}
__device__ __forceinline__ u32 pkmax(u32 a, u32 b) {
    return __builtin_bit_cast(u32, __builtin_elementwise_max(__builtin_bit_cast(u16x2, a), __builtin_bit_cast(u16x2, b)));
}
__device__ __forceinline__ bool encode(const u32 (&d)[16], u32 (&L)[8], u32 (&N)[4], u32& infoA, u32& infoB) {
    u32 m = d[0] & 0x7FFF7FFFu;
#pragma unroll
    for (int i = 1; i < 16; ++i) m = pkmax(m, d[i] & 0x7FFF7FFFu);
    const u32 mx = wave_umax(max(m & 0xFFFFu, m >> 16)) >> 8;
    const u32 base = mx > 7u ? mx - 7u : 0u;
    const u32 rep = base * 0x01010101u;
    u32 H[8], U[8], c[8], all = 0xFFFFFFFFu;
#pragma unroll
    for (int j = 0; j < 8; ++j) {
        L[j] = perm(d[2 * j + 1], d[2 * j], 0x06040200u);
        H[j] = perm(d[2 * j + 1], d[2 * j], 0x07050301u);
        U[j] = (H[j] | 0x80808080u) - rep;
        all &= U[j];
        c[j] = (U[j] & 0x07070707u) | ((H[j] >> 4) & 0x08080808u);
    }
#pragma unroll
    for (int j = 0; j < 4; ++j) N[j] = c[2 * j] | (c[2 * j + 1] << 4);
    infoA = base; infoB = 0;
    u64 miss = __ballot((all & 0x80808080u) != 0x80808080u);
    if (miss == 0) return true;
    u32 n = 0, e0 = 0, e1 = 0;
    while (miss) {
        const int ln = __builtin_ctzll(miss); miss &= miss - 1;
#pragma unroll
        for (int j = 0; j < 8; ++j) {
            u32 mm = ~(u32)__builtin_amdgcn_readlane((int)U[j], ln) & 0x80808080u;
            const u32 hj = (u32)__builtin_amdgcn_readlane((int)H[j], ln);
            while (mm) {
                const u32 b = (u32)__builtin_ctz(mm) >> 3; mm &= mm - 1;
                const u32 e = ((4u * j + b) * 64u + (u32)ln) | (((hj >> (8u * b)) & 0xFFu) << 11);
                if (n == 0) e0 = e; else if (n == 1) e1 = e;
                ++n;
            }
        }
        if (n > 2) break;
    }
    if (n > 2) { infoA = 0x80u; return false; }
    infoA = base | (n << 8) | (e0 << 10); infoB = e1;
    return true;
}
__device__ __forceinline__ void decode(const u32 (&L)[8], const u32 (&N)[4], u32 infoA, u32 infoB, int lane, u32 (&d)[16]) {
    const u32 rep = (infoA & 0x7Fu) * 0x01010101u;
    u32 hb[8];
#pragma unroll
    for (int j = 0; j < 4; ++j) {
        hb[2 * j] = ((N[j] & 0x07070707u) + rep) | ((N[j] & 0x08080808u) << 4);
        hb[2 * j + 1] = (((N[j] >> 4) & 0x07070707u) + rep) | (N[j] & 0x80808080u);
    }
    const u32 n = (infoA >> 8) & 3u;
    if (n) {
        for (u32 i = 0; i < n; ++i) {
            const u32 e = i == 0 ? (infoA >> 10) : infoB;
            const u32 idx = e & 0x7FFu, val = (e >> 11) & 0xFFu, ln = idx & 63u, v = idx >> 6, sh = (v & 3u) * 8u;
#pragma unroll
            for (int j = 0; j < 8; ++j)
                if ((v >> 2) == (u32)j && (u32)lane == ln) hb[j] = (hb[j] & ~(0xFFu << sh)) | (val << sh);
        }
    }
#pragma unroll
    for (int j = 0; j < 8; ++j) { d[2 * j] = perm(hb[j], L[j], 0x05010400u); d[2 * j + 1] = perm(hb[j], L[j], 0x07030602u); }
}
__device__ __forceinline__ void load_chunk(const char* base, u32 infoA, u32 infoB, int lane, u32 (&d)[16]) {
    u32x4 a, b, c, e;
    if (infoA & 0x80u) {
        ld4_sys(a, b, c, e, base, (u32)lane * 16u);
        d[0] = a.x; d[1] = a.y; d[2] = a.z; d[3] = a.w; d[4] = b.x; d[5] = b.y; d[6] = b.z; d[7] = b.w;
        d[8] = c.x; d[9] = c.y; d[10] = c.z; d[11] = c.w; d[12] = e.x; d[13] = e.y; d[14] = e.z; d[15] = e.w;
    } else {
        ld3_sys(a, b, c, base, (u32)lane * 16u);
        const u32 L[8] = {a.x, a.y, a.z, a.w, b.x, b.y, b.z, b.w};
        const u32 N[4] = {c.x, c.y, c.z, c.w};
        decode(L, N, infoA, infoB, lane, d);
    }
}

struct InItem { int isx, s, blk, j; };
__device__ __forceinline__ InItem in_item(int pos) {
    InItem it;
    if (pos < 384) {
        it.s = pos / 96; const int r = pos - it.s * 96;
        if (r < 32) { it.isx = 0; it.blk = r >> 2; it.j = r & 3; }
        else { it.isx = 1; it.blk = (r - 32) >> 2; it.j = r & 3; }
    } else { const int q = pos - 384; it.s = q >> 5; it.isx = 0; it.blk = 8 + ((q & 31) >> 2); it.j = q & 3; }
    return it;
}
__device__ __forceinline__ void in_addr(const Params& P, int dst, const InItem& it, int lane, const char*& sbase, u32 (&vo)[4]) {
    const int dm = (dst >> 2) & 1, dn = (dst >> 1) & 1, dk = dst & 1;
    if (it.isx) {
        const int owner = (it.blk & 1) * 4 + it.j;
        const int row = owner * 512 + dm * 256 + (it.blk >> 1) * 32;
        sbase = P.x + (size_t)row * 1024 + (size_t)(dk * 256 + it.s * 64) * 2;
        const u32 l = (u32)(lane & 15) * 1024u + (u32)(lane >> 4) * 16u;
        vo[0] = l; vo[1] = l + 64u; vo[2] = l + 16384u; vo[3] = l + 16448u;
    } else {
        const int k0 = dk * 256 + it.s * 64 + (it.j & 1) * 32, n0 = dn * 2048 + it.blk * 128 + (it.j >> 1) * 64;
        sbase = P.w + (size_t)k0 * 8192 + (size_t)n0 * 2;
        const u32 t = (u32)lane >> 5, g = ((u32)lane >> 3) & 3u, q = ((u32)lane >> 1) & 3u, u = (u32)lane & 1u;
        const u32 l = (8u * g + 4u * t + q) * 8192u + u * 64u;
        vo[0] = l; vo[1] = l + 16u; vo[2] = l + 32u; vo[3] = l + 48u;
    }
}
__device__ __forceinline__ void to_image(const u32x4 (&r)[4], int isx, u32 (&d)[16]) {
    if (isx) {
        d[0] = r[0].x; d[1] = r[0].y; d[2] = r[0].z; d[3] = r[0].w; d[4] = r[1].x; d[5] = r[1].y; d[6] = r[1].z; d[7] = r[1].w;
        d[8] = r[2].x; d[9] = r[2].y; d[10] = r[2].z; d[11] = r[2].w; d[12] = r[3].x; d[13] = r[3].y; d[14] = r[3].z; d[15] = r[3].w;
    } else {
        d[0] = r[0].x; d[1] = r[0].y; d[2] = r[2].x; d[3] = r[2].y; d[4] = r[0].z; d[5] = r[0].w; d[6] = r[2].z; d[7] = r[2].w;
        d[8] = r[1].x; d[9] = r[1].y; d[10] = r[3].x; d[11] = r[3].y; d[12] = r[1].z; d[13] = r[1].w; d[14] = r[3].z; d[15] = r[3].w;
    }
}
__device__ __forceinline__ size_t img_off(const InItem& it, int t) {
    const int j = it.isx ? it.j : (((it.j >> 1) & 1) * 2 + (it.j & 1));
    return (it.isx ? OFF_IMGA : OFF_IMGB) + ((size_t)(it.blk * NS + t) * 4 + j) * 4096;
}
__device__ __forceinline__ size_t rdy_off(const InItem& it, int t) { return OFF_RDY + ((size_t)((it.isx ? 0 : 64) + it.blk * 4 + it.j) * 32 + t) * 4; }
__device__ __forceinline__ void store_image(char* myws, const InItem& it, int t, const u32 (&d)[16], int lane, u32 call) {
    const char* ib = myws + img_off(it, t);
    const u32 vo = (u32)lane * 16u;
    st_agt<0>(ib, vo, (u32x4){d[0], d[1], d[2], d[3]});
    st_agt<1024>(ib, vo, (u32x4){d[4], d[5], d[6], d[7]});
    st_agt<2048>(ib, vo, (u32x4){d[8], d[9], d[10], d[11]});
    st_agt<3072>(ib, vo, (u32x4){d[12], d[13], d[14], d[15]});
    WAITV(0);
    if (lane == 0) __hip_atomic_store((GAS u32*)(myws + rdy_off(it, t)), call, __ATOMIC_RELAXED, __HIP_MEMORY_SCOPE_AGENT);
}

__device__ __forceinline__ void put_hdr_sys(const char* hp, u32 call, u32 a, u32 b, int lane) {
    if (lane == 0) st_sys<0>(hp, 0u, (u32x4){call, a, call, b});
}
__device__ __forceinline__ void pusher(const Params& P, Ctx& c, int cw, int wv, int lane, LAS u32x4* slots) {
    const int j = cw % 7 + 1, dst = (P.rank + j) & 7, idx = (cw / 7) * 2 + wv;
    char* myws = P.ws[P.rank];
    char* dws = P.ws[dst];
    const char* din = dws + OFF_IN + (size_t)P.rank * NPOS_IN * 4096;
    const char* dhin = dws + OFF_HIN + (size_t)P.rank * NPOS_IN * 16;
    const char* dp = dws + OFF_P + (size_t)P.rank * NPOS_OUT * 4096;
    const char* dhp = dws + OFF_HP + (size_t)P.rank * NPOS_OUT * 16;
    const char* q = myws + OFF_Q + (size_t)dst * NPOS_OUT * 4096;
    const u32 lo = (u32)lane * 16u;
    constexpr int NPOS = NPOS_IN + NPOS_OUT;
    const u32 slot0 = (u32)(size_t)slots + (u32)wv * 256u;

    u32x4 A[4], B[4];
    u32 hA = 0, hB = 0;
    int pos = idx;
    {
        const InItem it = in_item(pos); const char* sb; u32 vo[4];
        in_addr(P, dst, it, lane, sb, vo);
        ld4_issue(A, sb, vo[0], vo[1], vo[2], vo[3]);
        WAITV(0); PIN4(A);
    }
    const char* prev_h = nullptr; u32 prevA = 0, prevB = 0; bool have_prev = false;
    for (;;) {
        const int nxt = pos + PUSH_PER_DST;
        const bool have_next = nxt < NPOS;
        bool ready = have_next;
        u32 nA = 0, nB = 0;
        if (have_next && nxt >= NPOS_IN) {
            const u32x4 s = lds_poll16(slot0 + (u32)((nxt - NPOS_IN) >> 4) * 16u);
            const u32 s0 = rfl(s.x), s1 = rfl(s.z);
            ready = (s0 >= c.call && s1 >= c.call);
            nA = rfl(s.y); nB = rfl(s.w);
        }
        {
            const char* sb = q; u32 vo[4] = {lo, lo + 1024u, lo + 2048u, lo + 3072u};
            if (ready) {
                if (nxt < NPOS_IN) in_addr(P, dst, in_item(nxt), lane, sb, vo);
                else sb = q + (size_t)(nxt - NPOS_IN) * 4096;
            }
            ld4_issue(B, sb, vo[0], vo[1], vo[2], vo[3]);
        }
        const char* db; const char* hp; u32 ia, ib;
        if (pos < NPOS_IN) {
            const InItem it = in_item(pos);
            u32 d[16], L[8], N[4];
            to_image(A, it.isx, d);
            db = din + (size_t)pos * 4096; hp = dhin + (size_t)pos * 16;
            if (encode(d, L, N, ia, ib)) {
                st_sys<0>(db, lo, (u32x4){L[0], L[1], L[2], L[3]});
                st_sys<1024>(db, lo, (u32x4){L[4], L[5], L[6], L[7]});
                st_sys<2048>(db, lo, (u32x4){N[0], N[1], N[2], N[3]});
            } else {
                st_sys<0>(db, lo, (u32x4){d[0], d[1], d[2], d[3]});
                st_sys<1024>(db, lo, (u32x4){d[4], d[5], d[6], d[7]});
                st_sys<2048>(db, lo, (u32x4){d[8], d[9], d[10], d[11]});
                st_sys<3072>(db, lo, (u32x4){d[12], d[13], d[14], d[15]});
            }
        } else {
            db = dp + (size_t)(pos - NPOS_IN) * 4096; hp = dhp + (size_t)(pos - NPOS_IN) * 16;
            ia = hA; ib = hB;
            st_sys<0>(db, lo, A[0]); st_sys<1024>(db, lo, A[1]); st_sys<2048>(db, lo, A[2]);
            if (ia & 0x80u) st_sys<3072>(db, lo, A[3]);
        }
        if (ready) {
            WAITV(7);
            if (have_prev) { put_hdr_sys(prev_h, c.call, prevA, prevB, lane); WAITV(4); }
            else { WAITV(3); }
            PIN4(B);
            A[0] = B[0]; A[1] = B[1]; A[2] = B[2]; A[3] = B[3];
            hA = nA; hB = nB;
            prev_h = hp; prevA = ia; prevB = ib; have_prev = true;
            pos = nxt;
        } else {
            WAITV(0); PIN4(B);
            if (have_prev) put_hdr_sys(prev_h, c.call, prevA, prevB, lane);
            put_hdr_sys(hp, c.call, ia, ib, lane);
            have_prev = false;
            if (!have_next) break;
            const u64 t0 = rt(); u32 polls = 0; bool ok = false;
            while (!c.dead) {
                const u32x4 s = lds_poll16(slot0 + (u32)((nxt - NPOS_IN) >> 4) * 16u);
                if (rfl(s.x) >= c.call && rfl(s.z) >= c.call) { hA = rfl(s.y); hB = rfl(s.w); ok = true; break; }
                if (give_up(c, t0, polls)) break;
            }
            if (!ok) break;
            ld4_issue(A, q + (size_t)(nxt - NPOS_IN) * 4096, lo, lo + 1024u, lo + 2048u, lo + 3072u);
            WAITV(0); PIN4(A);
            pos = nxt;
        }
        if (pos == NPOS_IN + idx) STAMP(1);
    }
    STAMP(2);
}
__device__ __forceinline__ void scout(const Params& P, Ctx& c, int cw, int lane, LAS u32x4* slots) {
    const int j = cw % 7 + 1, dst = (P.rank + j) & 7;
    const int l = lane & 31, wv = l >> 4, k = l & 15;
    const int cidx = (cw / 7) * 2 + wv + PUSH_PER_DST * k;
    const char* hq = P.ws[P.rank] + OFF_HQ + (size_t)dst * NPOS_OUT * 16;
    const u32 vo = (u32)cidx * 16u;
    bool done = false;
    const u64 t0 = rt(); u32 polls = 0;
    for (;;) {
        u32x4 h;
        asm volatile("s_nop 4\n\tglobal_load_dwordx4 %0, %1, %2 sc1\n\ts_waitcnt vmcnt(0)" : "=&v"(h) : "v"(vo), "s"(uni(hq)) : "memory");
        if (!done && h.x >= c.call && h.z >= c.call) { if (lane < 32) slots[l] = h; done = true; }
        if (__all(done)) break;
        __builtin_amdgcn_s_sleep(8);
        if (give_up(c, t0, polls)) break;
    }
}
__device__ __forceinline__ void unpacker(const Params& P, Ctx& c, int u, int lane) {
    char* myws = P.ws[P.rank];
    if (u < NPOS_IN) {
        const InItem it = in_item(u); const char* sb; u32 vo[4]; u32x4 r[4]; u32 d[16];
        in_addr(P, P.rank, it, lane, sb, vo);
        ld4_issue(r, sb, vo[0], vo[1], vo[2], vo[3]);
        WAITV(0); PIN4(r);
        to_image(r, it.isx, d);
        store_image(myws, it, it.s, d, lane, c.call);
    }
    STAMP(1);
    for (int q = (u + NUNP - NPOS_IN) % NUNP; q < NPOS_IN * 7; q += NUNP) {
        const int pos = q / 7, rel = q - pos * 7 + 1;
        const int src = (P.rank + rel) & 7;
        u32 ia, ib;
        if (!wait_hdr(c, myws + OFF_HIN + ((size_t)src * NPOS_IN + pos) * 16, ia, ib)) break;
        const InItem it = in_item(pos);
        u32 d[16];
        load_chunk(myws + OFF_IN + ((size_t)src * NPOS_IN + pos) * 4096, ia, ib, lane, d);
        store_image(myws, it, 4 + it.s * 7 + (rel - 1), d, lane, c.call);
    }
    STAMP(2);
}

struct Frags { s16x8 a[2][2]; s16x4 bl[4][2], bh[4][2]; };
template <int OFF> __device__ __forceinline__ void dsr128(s16x8& r, u32 addr) { asm volatile("ds_read_b128 %0, %1 offset:%2" : "=v"(r) : "v"(addr), "i"(OFF)); }
template <int OFF> __device__ __forceinline__ void dsrtr(s16x4& r, u32 addr) { asm volatile("ds_read_b64_tr_b16 %0, %1 offset:%2" : "=v"(r) : "v"(addr), "i"(OFF)); }
__device__ __forceinline__ void mf(f32x4& acc, const s16x4& bl, const s16x4& bh, const s16x8& a) {
    s16x8 b = __builtin_shufflevector(bl, bh, 0, 1, 2, 3, 4, 5, 6, 7);
    asm volatile("v_mfma_f32_16x16x32_bf16 %0, %1, %2, %0" : "+v"(acc) : "v"(b), "v"(a));
}
template <int OFF> __device__ __forceinline__ void dma(const char* src, LAS char* dst) {
    __builtin_amdgcn_global_load_lds((const GAS void*)src, (LAS void*)dst, 16, OFF, 16  );
}
__device__ __forceinline__ void wait_frags(Frags& f) {
    asm volatile("s_waitcnt lgkmcnt(0)" : "+v"(f.a[0][0]), "+v"(f.a[0][1]), "+v"(f.a[1][0]), "+v"(f.a[1][1]),
        "+v"(f.bl[0][0]), "+v"(f.bl[1][0]), "+v"(f.bl[2][0]), "+v"(f.bl[3][0]), "+v"(f.bl[0][1]), "+v"(f.bl[1][1]), "+v"(f.bl[2][1]), "+v"(f.bl[3][1]) :: "memory");
    asm volatile("" : "+v"(f.bh[0][0]), "+v"(f.bh[1][0]), "+v"(f.bh[2][0]), "+v"(f.bh[3][0]), "+v"(f.bh[0][1]), "+v"(f.bh[1][1]), "+v"(f.bh[2][1]), "+v"(f.bh[3][1]));
}
__device__ __forceinline__ void read_all(Frags& n, u32 aa, u32 ba) {
    dsr128<0>(n.a[0][0], aa); dsr128<2048>(n.a[1][0], aa);
    dsrtr<0>(n.bl[0][0], ba); dsrtr<512>(n.bh[0][0], ba); dsrtr<1024>(n.bl[1][0], ba); dsrtr<1536>(n.bh[1][0], ba);
    dsrtr<2048>(n.bl[2][0], ba); dsrtr<2560>(n.bh[2][0], ba); dsrtr<3072>(n.bl[3][0], ba); dsrtr<3584>(n.bh[3][0], ba);
    dsr128<1024>(n.a[0][1], aa); dsr128<3072>(n.a[1][1], aa);
    dsrtr<4096>(n.bl[0][1], ba); dsrtr<4608>(n.bh[0][1], ba); dsrtr<5120>(n.bl[1][1], ba); dsrtr<5632>(n.bh[1][1], ba);
    dsrtr<6144>(n.bl[2][1], ba); dsrtr<6656>(n.bh[2][1], ba); dsrtr<7168>(n.bl[3][1], ba); dsrtr<7680>(n.bh[3][1], ba);
}
__device__ __forceinline__ void kstep(f32x4 (&acc)[2][4], const Frags& c, Frags& n, u32 aa, u32 ba, const char* src, LAS char* dst) {
#define MF(mi, ni, kk) mf(acc[mi][ni], c.bl[ni][kk], c.bh[ni][kk], c.a[mi][kk])
#define RA(mi, kk) dsr128<(mi * 2 + kk) * 1024>(n.a[mi][kk], aa)
#define RB(ni, kk) do { dsrtr<kk * 4096 + ni * 1024>(n.bl[ni][kk], ba); dsrtr<kk * 4096 + ni * 1024 + 512>(n.bh[ni][kk], ba); } while (0)
    MF(0,0,0); RA(0,0); RA(1,0);
    MF(0,1,0); RB(0,0);
    MF(0,2,0); RB(1,0);
    MF(0,3,0); RB(2,0);
    MF(1,0,0); RB(3,0);
    MF(1,1,0); RA(0,1); RA(1,1);
    MF(1,2,0); RB(0,1);
    MF(1,3,0); RB(1,1);
    MF(0,0,1); RB(2,1);
    MF(0,1,1); RB(3,1);
    MF(0,2,1); dma<0>(src, dst);
    MF(0,3,1); dma<1024>(src, dst);
    MF(1,0,1); dma<2048>(src, dst);
    MF(1,1,1); dma<3072>(src, dst);
    MF(1,2,1);
    MF(1,3,1);
#undef MF
#undef RA
#undef RB
}
__device__ __forceinline__ void need_step(Ctx& c, const char* rdyline, u32& mask, int t, int lane) {
    if ((mask >> t) & 1u) return;
    if (c.dead) return;
    const u64 t0 = rt(); u32 polls = 0;
    for (;;) {
        u32 v; const u32 vo = (u32)(lane & 31) * 4u;
        asm volatile("s_nop 4\n\tglobal_load_dword %0, %1, %2 sc1\n\ts_waitcnt vmcnt(0)" : "=&v"(v) : "v"(vo), "s"(uni(rdyline)) : "memory");
        mask = (u32)__ballot(v >= c.call);
        if ((mask >> t) & 1u) return;
        if (give_up(c, t0, polls)) return;
    }
}
__device__ __forceinline__ u32 cvt_pk_bf16(float lo, float hi) { u32 r; asm volatile("v_cvt_pk_bf16_f32 %0, %1, %2" : "=v"(r) : "v"(lo), "v"(hi)); return r; }

__device__ __forceinline__ void gemm_tile(const Params& P, Ctx& c, LAS char* lds, int rb, int cb, int w, int lane) {
    char* myws = P.ws[P.rank];
    const int wm = w & 3, wn = w >> 2;
    const u32 ldsbase = (u32)(size_t)lds;
    const char* src = myws + (w < 4 ? OFF_IMGA + ((size_t)rb * NS * 4 + w) * 4096 : OFF_IMGB + ((size_t)cb * NS * 4 + (w - 4)) * 4096) + lane * 16;
    const char* rdyline = myws + OFF_RDY + (size_t)(w < 4 ? rb * 4 + w : 64 + cb * 4 + (w - 4)) * 128;
    const u32 dstw = (u32)w * 4096u;
    const u32 aoff = ldsbase + wm * 4096 + lane * 16, boff = ldsbase + 16384 + wn * 8192 + lane * 8;
    u32 mask = 0;
    f32x4 acc[2][4];
#pragma unroll
    for (int mi = 0; mi < 2; ++mi)
#pragma unroll
        for (int ni = 0; ni < 4; ++ni) acc[mi][ni] = (f32x4){0.f, 0.f, 0.f, 0.f};
    Frags f0, f1;
    need_step(c, rdyline, mask, 3, lane); need_step(c, rdyline, mask, 0, lane); need_step(c, rdyline, mask, 1, lane); need_step(c, rdyline, mask, 2, lane);
#pragma unroll
    for (int s = 0; s < 4; ++s) {
        LAS char* d = lds + s * 32768 + dstw; const char* g = src + (size_t)s * 16384;
        dma<0>(g, d); dma<1024>(g, d); dma<2048>(g, d); dma<3072>(g, d);
    }
    WAITV(12); BAR();
    read_all(f0, aoff, boff);
    wait_frags(f0);
    asm volatile("s_nop 1");
    for (int s = 0; s < NS; s += 2) {
        const int n4 = s + 4 < NS ? s + 4 : s + 4 - NS, n5 = s + 5 < NS ? s + 5 : s + 5 - NS;
        WAITV(8); BAR(); SCHED();
        need_step(c, rdyline, mask, n4, lane);
        kstep(acc, f0, f1, aoff + ((s + 1) & 3) * 32768, boff + ((s + 1) & 3) * 32768, src + (size_t)n4 * 16384, lds + (s & 3) * 32768 + dstw);
        wait_frags(f1); SCHED();
        WAITV(8); BAR(); SCHED();
        need_step(c, rdyline, mask, n5, lane);
        kstep(acc, f1, f0, aoff + ((s + 2) & 3) * 32768, boff + ((s + 2) & 3) * 32768, src + (size_t)n5 * 16384, lds + ((s + 1) & 3) * 32768 + dstw);
        wait_frags(f0); SCHED();
    }
    WAITV(0); BAR();
    asm volatile("s_nop 15\n\ts_nop 15" ::: "memory");
    asm volatile("" : "+v"(acc[0][0]), "+v"(acc[0][1]), "+v"(acc[0][2]), "+v"(acc[0][3]), "+v"(acc[1][0]), "+v"(acc[1][1]), "+v"(acc[1][2]), "+v"(acc[1][3]));
    u32 d[16], L[8], N[4], ia, ib;
#pragma unroll
    for (int mi = 0; mi < 2; ++mi)
#pragma unroll
        for (int ni = 0; ni < 4; ++ni) {
            d[mi * 8 + ni * 2] = cvt_pk_bf16(acc[mi][ni][0], acc[mi][ni][1]);
            d[mi * 8 + ni * 2 + 1] = cvt_pk_bf16(acc[mi][ni][2], acc[mi][ni][3]);
        }
    const int owner = (rb & 1) * 4 + wm;
    const int cidx = (cb >> 3) * 128 + (rb >> 1) * 16 + (cb & 7) * 2 + wn;
    const bool mine = owner == P.rank;
    const char* db = myws + (mine ? OFF_P : OFF_Q) + ((size_t)owner * NPOS_OUT + cidx) * 4096;
    const char* hp = myws + (mine ? OFF_HP : OFF_HQ) + ((size_t)owner * NPOS_OUT + cidx) * 16;
    const u32 lo = (u32)lane * 16u;
    if (encode(d, L, N, ia, ib)) {
        st_agt<0>(db, lo, (u32x4){L[0], L[1], L[2], L[3]});
        st_agt<1024>(db, lo, (u32x4){L[4], L[5], L[6], L[7]});
        st_agt<2048>(db, lo, (u32x4){N[0], N[1], N[2], N[3]});
    } else {
        st_agt<0>(db, lo, (u32x4){d[0], d[1], d[2], d[3]});
        st_agt<1024>(db, lo, (u32x4){d[4], d[5], d[6], d[7]});
        st_agt<2048>(db, lo, (u32x4){d[8], d[9], d[10], d[11]});
        st_agt<3072>(db, lo, (u32x4){d[12], d[13], d[14], d[15]});
    }
    WAITV(0);
    if (!c.dead && lane == 0) st_agt<0>(hp, 0u, (u32x4){c.call, ia, c.call, ib});
}

__device__ __forceinline__ float bf_lo(u32 v) { return __uint_as_float(v << 16); }
__device__ __forceinline__ float bf_hi(u32 v) { return __uint_as_float(v & 0xFFFF0000u); }

__device__ __forceinline__ void gemm_role(const Params& P, Ctx& c, LAS char* lds, int g, int w, int lane) {
    char* myws = P.ws[P.rank];
    const int xcd = g & 7, i = g >> 3;
    const int rb = (xcd & 3) * 4 + (i & 3), cbl = (xcd >> 2) * 4 + (i >> 2);
    STAMP(0);
    gemm_tile(P, c, lds, rb, cbl, w, lane);
    STAMP(1);
    gemm_tile(P, c, lds, rb, 8 + cbl, w, lane);
    STAMP(2);
    const int quad = w >> 1, cidx = (w & 1) * 128 + g;
    const int qm = quad >> 1, qn = quad & 1;
    const int s0 = (qm << 2) | (qn << 1), s1 = s0 | 1;
    float y[32];
    {
        u32 a0, b0, a1, b1;
        const bool ok0 = wait_hdr(c, myws + OFF_HP + ((size_t)s0 * NPOS_OUT + cidx) * 16, a0, b0);
        const bool ok1 = wait_hdr(c, myws + OFF_HP + ((size_t)s1 * NPOS_OUT + cidx) * 16, a1, b1);
        u32 d0[16], d1[16];
        if (ok0 && ok1) {
            load_chunk(myws + OFF_P + ((size_t)s0 * NPOS_OUT + cidx) * 4096, a0, b0, lane, d0);
            load_chunk(myws + OFF_P + ((size_t)s1 * NPOS_OUT + cidx) * 4096, a1, b1, lane, d1);
        } else {
#pragma unroll
            for (int k = 0; k < 16; ++k) { d0[k] = 0; d1[k] = 0; }
        }
#pragma unroll
        for (int k = 0; k < 16; ++k) { y[2 * k] = bf_lo(d0[k]) + bf_lo(d1[k]); y[2 * k + 1] = bf_hi(d0[k]) + bf_hi(d1[k]); }
    }
    STAMP(3);
    u32 mx = 0;
#pragma unroll
    for (int k = 0; k < 32; ++k) mx = max(mx, __float_as_uint(y[k]) & 0x7FFFFFFFu);
    mx = wave_umax(mx);
    LAS u32* red = (LAS u32*)(lds + 131072);
    if (lane == 0) red[w] = mx;
    __syncthreads();
    if (w == 0 && lane < NR) {
        u32 m = 0;
#pragma unroll
        for (int k = 0; k < 8; ++k) m = max(m, red[k]);
        if (!c.dead)
            __hip_atomic_store((GAS u64*)(P.ws[lane] + OFF_AMAX + ((size_t)P.rank * NGEMM + g) * 8), ((u64)m << 32) | c.call, __ATOMIC_RELAXED, __HIP_MEMORY_SCOPE_SYSTEM);
    }
    u32 gm = 0;
    if (!c.dead) {
        const u64 t0 = rt(); u32 polls = 0;
        const char* ab = myws + OFF_AMAX; const u32 vo = (u32)(w * 64 + lane) * 16u;
        for (;;) {
            u32x4 h;
            asm volatile("s_nop 4\n\tglobal_load_dwordx4 %0, %1, %2 sc0 sc1\n\ts_waitcnt vmcnt(0)" : "=&v"(h) : "v"(vo), "s"(uni(ab)) : "memory");
            if (__all(h.x >= c.call && h.z >= c.call)) { gm = max(h.y, h.w); break; }
            if (give_up(c, t0, polls)) break;
        }
    }
    gm = wave_umax(gm);
    if (lane == 0) red[8 + w] = gm;
    __syncthreads();
    u32 am = 0;
#pragma unroll
    for (int k = 0; k < 8; ++k) am = max(am, red[8 + k]);
    STAMP(4);
    const u32 tm = __hip_atomic_load((const GAS u32*)c.tmo, __ATOMIC_RELAXED, __HIP_MEMORY_SCOPE_AGENT);
    const bool bad = c.dead || tm == c.call;
    const float scale = __uint_as_float(am) / 448.0f;
    const float inv = 1.0f / scale;
    const int ph = cidx >> 7, rbh = (cidx >> 4) & 7, cb7 = (cidx >> 1) & 7, wn = cidx & 1;
    char* orow = P.out + (size_t)(qm * 256 + rbh * 32 + (lane & 15)) * 4096 + qn * 2048 + (ph * 8 + cb7) * 128 + wn * 64 + (lane >> 4) * 16;
#pragma unroll
    for (int mi = 0; mi < 2; ++mi) {
        u32x4 o;
#pragma unroll
        for (int k = 0; k < 4; ++k) {
            const float* v = y + mi * 16 + k * 4;
            int p = __builtin_amdgcn_cvt_pk_fp8_f32(v[0] * inv, v[1] * inv, 0, false);
            p = __builtin_amdgcn_cvt_pk_fp8_f32(v[2] * inv, v[3] * inv, p, true);
            o[k] = bad ? 0x7F7F7F7Fu : (u32)p;
        }
        *(u32x4*)(orow + (size_t)mi * 16 * 4096) = o;
    }
    if (g == 0 && w == 0 && lane == 0) *(u32*)(myws + OFF_CTL) = c.call;
    STAMP(5);
}

__global__ void __launch_bounds__(512, 2) gemm_rs_kernel(Params P) {
    extern __shared__ __attribute__((aligned(16))) char smem[];
    LAS char* lds = (LAS char*)smem;
    const int tid = threadIdx.x, w = (int)rfl((u32)tid >> 6), lane = tid & 63;
    char* myws = P.ws[P.rank];
    Ctx c;
    c.tmo = myws + OFF_CTL + 128;
    c.call = rfl(__hip_atomic_load((const GAS u32*)(myws + OFF_CTL), __ATOMIC_RELAXED, __HIP_MEMORY_SCOPE_AGENT)) + 1u;
    c.dead = false;
    const int b = blockIdx.x;
    if (b < NGEMM) { gemm_role(P, c, lds, b, w, lane); return; }
    const int cw = b - NGEMM;
    STAMP(0);
    if (cw < NPUSHWG) {
        LAS u32x4* slots = (LAS u32x4*)lds;
        if (tid < 32) slots[tid] = (u32x4){0u, 0u, 0u, 0u};
        __syncthreads();
        if (w < 2) pusher(P, c, cw, w, lane, slots);
        else if (w == 2) scout(P, c, cw, lane, slots);
        else unpacker(P, c, cw * 5 + (w - 3), lane);
    } else {
        unpacker(P, c, NPUSHWG * 5 + (cw - NPUSHWG) * 8 + w, lane);
    }
}

}

extern "C" void kernel_launch(int rank, int nranks, ncclComm_t comm, hipStream_t stream,
                              void* const* d_in, const int* in_sizes, int n_in,
                              void* d_out, int out_size,
                              const int* in_dtypes, int out_dtype,
                              void* const* d_out_all,
                              void* const* d_ws_all, size_t ws_bytes) {
    (void)comm; (void)in_dtypes; (void)out_dtype; (void)d_out_all;
    if (nranks != gr::NR || (unsigned)rank >= (unsigned)gr::NR || n_in < 2 || ws_bytes < gr::WS_NEED ||
        in_sizes[0] != 4096 * 512 || in_sizes[1] != 512 * 4096 || out_size != 512 * 4096) {
        fprintf(stderr, "kernel_launch: unexpected shapes\n");
        return;
    }
    static bool attr_set = false;
    if (!attr_set) {
        hipError_t e = hipFuncSetAttribute((const void*)gr::gemm_rs_kernel, hipFuncAttributeMaxDynamicSharedMemorySize, gr::LDS_BYTES);
        if (e != hipSuccess) { fprintf(stderr, "hipFuncSetAttribute: %s\n", hipGetErrorString(e)); return; }
        attr_set = true;
    }
    gr::Params p;
    p.x = (const char*)d_in[0]; p.w = (const char*)d_in[1]; p.out = (char*)d_out;
    for (int i = 0; i < gr::NR; ++i) p.ws[i] = (char*)d_ws_all[i];
    p.rank = rank; p.pad = 0;
    gr::gemm_rs_kernel<<<dim3(gr::NGEMM + gr::NCOMM), dim3(512), gr::LDS_BYTES, stream>>>(p);
    hipError_t e = hipGetLastError();
    if (e != hipSuccess) fprintf(stderr, "launch: %s\n", hipGetErrorString(e));
}
```

```cpp
#include <hip/hip_runtime.h>
#include <rccl/rccl.h>
#include <cstdio>
#include <cstdint>

#define LAS __attribute__((address_space(3)))
#define GAS __attribute__((address_space(1)))

namespace gr {

typedef unsigned u32;
typedef unsigned long long u64;
typedef unsigned u32x4 __attribute__((ext_vector_type(4)));
typedef unsigned short u16x2 __attribute__((ext_vector_type(2)));
typedef short s16x4 __attribute__((ext_vector_type(4)));
typedef short s16x8 __attribute__((ext_vector_type(8)));
typedef float f32x4 __attribute__((ext_vector_type(4)));

constexpr int NR = 8;
constexpr int NS = 32;
constexpr int NRED = 128;
constexpr int NGEMM = 192;
#define GR_PWG 4
#define GR_PW 4
constexpr int PWG = GR_PWG;
constexpr int NPUSHWG = 7 * PWG;
constexpr int PW = GR_PW;
constexpr int PUSH_PER_DST = PWG * PW;
constexpr int NUNPWG = 256 - NGEMM - NPUSHWG;
constexpr int NUNP = NUNPWG * 8;
#define GR_DIRECT 1
constexpr bool DIRECT_B = GR_DIRECT;
constexpr int NPOS_IN = 512;
constexpr int NPOS_OUT = 256;
constexpr u64 DEADLINE = 40000000ull;

constexpr size_t OFF_CTL = 0;
constexpr size_t OFF_AMAX = 4096;
constexpr size_t OFF_RDY = 16384;
constexpr size_t OFF_HIN = 32768;
constexpr size_t OFF_HP = OFF_HIN + 65536;
constexpr size_t OFF_HQ = OFF_HP + 32768;
constexpr size_t OFF_DBG = 262144;
constexpr size_t OFF_IN = 1u << 20;
constexpr size_t OFF_IMGA = OFF_IN + (16u << 20);
constexpr size_t OFF_IMGB = OFF_IMGA + (8u << 20);
constexpr size_t OFF_P = OFF_IMGB + (8u << 20);
constexpr size_t OFF_Q = OFF_P + (8u << 20);
constexpr size_t WS_NEED = OFF_Q + (8u << 20);
#define GR_NST 5
constexpr int NST = GR_NST;
constexpr int LDS_BYTES = NST * 32768;

struct Params {
    const char* x;
    const char* w;
    char* out;
    char* ws[NR];
    int rank;
    int pad;
};

#define WAITV(n) asm volatile("s_waitcnt vmcnt(" #n ")" ::: "memory")
#define BAR() __builtin_amdgcn_s_barrier()
#define SCHED() __builtin_amdgcn_sched_barrier(0)

__device__ __forceinline__ u64 rt() { return __builtin_amdgcn_s_memrealtime(); }
__device__ __forceinline__ u32 rfl(u32 v) { return (u32)__builtin_amdgcn_readfirstlane((int)v); }

#define STAMP(slot) do { } while (0)
#define STAMPV(slot, val) do { } while (0)

__device__ __forceinline__ const char* uni(const char* p) {
    const u64 v = (u64)p;
    return (const char*)(((u64)rfl((u32)(v >> 32)) << 32) | (u64)rfl((u32)v));
}
template <int OFF> __device__ __forceinline__ void st_sys(const char* sbase, u32 voff, u32x4 v) {
    asm volatile("s_nop 4\n\tglobal_store_dwordx4 %0, %1, %2 offset:%3 sc0 sc1\n\ts_nop 1" :: "v"(voff), "v"(v), "s"(uni(sbase)), "i"(OFF) : "memory");
}
template <int OFF> __device__ __forceinline__ void st_agt(const char* sbase, u32 voff, u32x4 v) {
    asm volatile("s_nop 4\n\tglobal_store_dwordx4 %0, %1, %2 offset:%3 sc1\n\ts_nop 1" :: "v"(voff), "v"(v), "s"(uni(sbase)), "i"(OFF) : "memory");
}
__device__ __forceinline__ u32x4 poll16(const char* sbase) {
    u32x4 r; u32 z = 0;
    asm volatile("s_nop 4\n\tglobal_load_dwordx4 %0, %1, %2 sc0 sc1\n\ts_waitcnt vmcnt(0)" : "=&v"(r) : "v"(z), "s"(uni(sbase)) : "memory");
    return r;
}
__device__ __forceinline__ void ld3_sys(u32x4& a, u32x4& b, u32x4& c, const char* sbase, u32 voff) {
    asm volatile("s_nop 4\n\tglobal_load_dwordx4 %0, %3, %4 sc0 sc1\n\tglobal_load_dwordx4 %1, %3, %4 offset:1024 sc0 sc1\n\t"
                 "global_load_dwordx4 %2, %3, %4 offset:2048 sc0 sc1\n\ts_waitcnt vmcnt(0)"
                 : "=&v"(a), "=&v"(b), "=&v"(c) : "v"(voff), "s"(uni(sbase)) : "memory");
}
__device__ __forceinline__ void ld4_sys(u32x4& a, u32x4& b, u32x4& c, u32x4& d, const char* sbase, u32 voff) {
    asm volatile("s_nop 4\n\tglobal_load_dwordx4 %0, %4, %5 sc0 sc1\n\tglobal_load_dwordx4 %1, %4, %5 offset:1024 sc0 sc1\n\t"
                 "global_load_dwordx4 %2, %4, %5 offset:2048 sc0 sc1\n\tglobal_load_dwordx4 %3, %4, %5 offset:3072 sc0 sc1\n\ts_waitcnt vmcnt(0)"
                 : "=&v"(a), "=&v"(b), "=&v"(c), "=&v"(d) : "v"(voff), "s"(uni(sbase)) : "memory");
}
__device__ __forceinline__ void ld4_issue(u32x4 (&r)[4], const char* sbase, u32 v0, u32 v1, u32 v2, u32 v3) {
    asm volatile("s_nop 4\n\tglobal_load_dwordx4 %0, %4, %8 sc1\n\tglobal_load_dwordx4 %1, %5, %8 sc1\n\t"
                 "global_load_dwordx4 %2, %6, %8 sc1\n\tglobal_load_dwordx4 %3, %7, %8 sc1"
                 : "=&v"(r[0]), "=&v"(r[1]), "=&v"(r[2]), "=&v"(r[3]) : "v"(v0), "v"(v1), "v"(v2), "v"(v3), "s"(uni(sbase)) : "memory");
}
#define PIN4(r) asm volatile("" : "+v"(r[0]), "+v"(r[1]), "+v"(r[2]), "+v"(r[3]))

__device__ __forceinline__ u32x4 lds_poll16(u32 addr) {
    u32x4 r;
    asm volatile("ds_read_b128 %0, %1\n\ts_waitcnt lgkmcnt(0)" : "=&v"(r) : "v"(addr) : "memory");
    return r;
}
struct Ctx { const char* tmo; u32 call; bool dead;
};
__device__ __forceinline__ bool give_up(Ctx& c, u64 t0, u32& polls) {
    __builtin_amdgcn_s_sleep(2);
    ++polls;
    if ((polls & 15u) == 0u) {
        u32 t = __hip_atomic_load((const GAS u32*)c.tmo, __ATOMIC_RELAXED, __HIP_MEMORY_SCOPE_AGENT);
        if (rfl(t) == c.call) { c.dead = true; return true; }
        if (rt() - t0 > DEADLINE) {
            __hip_atomic_store((GAS u32*)c.tmo, c.call, __ATOMIC_RELAXED, __HIP_MEMORY_SCOPE_AGENT);
            c.dead = true; return true;
        }
    }
    return false;
}
__device__ __forceinline__ bool wait_hdr(Ctx& c, const char* hp, u32& a, u32& b) {
    a = 0; b = 0;
    if (c.dead) return false;
    const u64 t0 = rt(); u32 polls = 0;
    for (;;) {
        u32x4 h = poll16(hp);
        const u32 s0 = rfl(h.x), s1 = rfl(h.z);
        if (s0 >= c.call && s1 >= c.call) { a = rfl(h.y); b = rfl(h.w); return true; }
        if (give_up(c, t0, polls)) return false;
    }
}

__device__ __forceinline__ u32 perm(u32 s0, u32 s1, u32 sel) { return __builtin_amdgcn_perm(s0, s1, sel); }
__device__ __forceinline__ u32 wave_umax(u32 v) {
    v = max(v, (u32)__builtin_amdgcn_update_dpp(0, (int)v, 0x111, 0xf, 0xf, true));
    v = max(v, (u32)__builtin_amdgcn_update_dpp(0, (int)v, 0x112, 0xf, 0xf, true));
    v = max(v, (u32)__builtin_amdgcn_update_dpp(0, (int)v, 0x114, 0xf, 0xf, true));
    v = max(v, (u32)__builtin_amdgcn_update_dpp(0, (int)v, 0x118, 0xf, 0xf, true));
    v = max(v, (u32)__builtin_amdgcn_update_dpp(0, (int)v, 0x142, 0xa, 0xf, true));
    v = max(v, (u32)__builtin_amdgcn_update_dpp(0, (int)v, 0x143, 0xc, 0xf, true));
    return (u32)__builtin_amdgcn_readlane((int)v, 63);
}
__device__ __forceinline__ u32 pkmax(u32 a, u32 b) {
    return __builtin_bit_cast(u32, __builtin_elementwise_max(__builtin_bit_cast(u16x2, a), __builtin_bit_cast(u16x2, b)));
}
__device__ __forceinline__ u32 byte_of(u32 h0, u32 h1, u32 h2, u32 h3, u32 h4, u32 h5, u32 h6, u32 h7, u32 v) {
    asm volatile("" : "+v"(h0), "+v"(h1), "+v"(h2), "+v"(h3), "+v"(h4), "+v"(h5), "+v"(h6), "+v"(h7));
    const bool b0 = v & 4u, b1 = v & 8u, b2 = v & 16u;
    const u32 x0 = b0 ? h1 : h0, x1 = b0 ? h3 : h2, x2 = b0 ? h5 : h4, x3 = b0 ? h7 : h6;
    const u32 y0 = b1 ? x1 : x0, y1 = b1 ? x3 : x2;
    return ((b2 ? y1 : y0) >> ((v & 3u) * 8u)) & 0xFFu;
}
__device__ __forceinline__ void collect_escapes(const u32 (&U)[8], const u32 (&H)[8], u32 base, u32& infoA, u32& infoB) {
    u32 mk = 0;
#pragma unroll
    for (int j = 0; j < 8; ++j) mk |= ((((~U[j] >> 7) & 0x01010101u) * 0x01020408u) >> 24 & 15u) << (4 * j);
    const u32 lane = threadIdx.x & 63u, cnt = (u32)__builtin_popcount(mk);
    const u32 v1 = (u32)__builtin_ctz(mk | 0x80000000u), mk2 = mk & (mk - 1u), v2 = (u32)__builtin_ctz(mk2 | 0x80000000u);
    const u32 e1 = (v1 * 64u + lane) | (byte_of(H[0], H[1], H[2], H[3], H[4], H[5], H[6], H[7], v1) << 11), e2 = (v2 * 64u + lane) | (byte_of(H[0], H[1], H[2], H[3], H[4], H[5], H[6], H[7], v2) << 11);
    u64 any = __ballot(mk != 0u);
    const int l1 = __builtin_ctzll(any); any &= any - 1;
    const u32 c1 = (u32)__builtin_amdgcn_readlane((int)cnt, l1), a1 = (u32)__builtin_amdgcn_readlane((int)e1, l1), a2 = (u32)__builtin_amdgcn_readlane((int)e2, l1);
    u32 n = c1, f0 = a1, f1 = a2;
    if (any) {
        const int l2 = __builtin_ctzll(any); any &= any - 1;
        n += (u32)__builtin_amdgcn_readlane((int)cnt, l2);
        f1 = (u32)__builtin_amdgcn_readlane((int)e1, l2);
        if (any) n = 3;
    }
    if (n > 2) { infoA = 0x80u; infoB = 0; } else { infoA = base | (n << 8) | (f0 << 10); infoB = n > 1 ? f1 : 0u; }
}
__device__ __forceinline__ bool encode(const u32 (&d)[16], u32 (&L)[8], u32 (&N)[4], u32& infoA, u32& infoB) {
    u32 m[8];
#pragma unroll
    for (int i = 0; i < 8; ++i) m[i] = pkmax(d[2 * i] & 0x7FFF7FFFu, d[2 * i + 1] & 0x7FFF7FFFu);
    const u32 m4 = pkmax(pkmax(pkmax(m[0], m[1]), pkmax(m[2], m[3])), pkmax(pkmax(m[4], m[5]), pkmax(m[6], m[7])));
    const u32 mx = wave_umax(max(m4 & 0xFFFFu, m4 >> 16)) >> 8;
    const u32 base = mx > 7u ? mx - 7u : 0u;
    const u32 rep = base * 0x01010101u;
    u32 H[8], U[8], c[8];
#pragma unroll
    for (int j = 0; j < 8; ++j) {
        L[j] = perm(d[2 * j + 1], d[2 * j], 0x06040200u);
        H[j] = perm(d[2 * j + 1], d[2 * j], 0x07050301u);
        U[j] = (H[j] | 0x80808080u) - rep;
        c[j] = (U[j] & 0x07070707u) | ((H[j] >> 4) & 0x08080808u);
    }
    const u32 all = ((U[0] & U[1]) & (U[2] & U[3])) & ((U[4] & U[5]) & (U[6] & U[7]));
#pragma unroll
    for (int j = 0; j < 4; ++j) N[j] = c[2 * j] | (c[2 * j + 1] << 4);
    infoA = base; infoB = 0;
    if (__ballot((all & 0x80808080u) != 0x80808080u) != 0) {
        collect_escapes(U, H, base, infoA, infoB);
    }
    return !(infoA & 0x80u);
}
__device__ __forceinline__ void patch_escape(u32 (&hb)[8], u32 e) {
    const u32 idx = e & 0x7FFu, val = (e >> 11) & 0xFFu, ln = idx & 63u, v = idx >> 6, sh = (v & 3u) * 8u, jj = v >> 2;
    const bool me = (threadIdx.x & 63u) == ln;
    const u32 keep = me ? ~(0xFFu << sh) : 0xFFFFFFFFu, ins = me ? val << sh : 0u;
#pragma unroll
    for (int j = 0; j < 8; ++j) hb[j] = jj == (u32)j ? (hb[j] & keep) | ins : hb[j];
}
__device__ __forceinline__ void decode(const u32 (&L)[8], const u32 (&N)[4], u32 infoA, u32 infoB, u32 (&d)[16]) {
    const u32 rep = (infoA & 0x7Fu) * 0x01010101u;
    u32 hb[8];
#pragma unroll
    for (int j = 0; j < 4; ++j) {
        hb[2 * j] = ((N[j] & 0x07070707u) + rep) | ((N[j] & 0x08080808u) << 4);
        hb[2 * j + 1] = (((N[j] >> 4) & 0x07070707u) + rep) | (N[j] & 0x80808080u);
    }
    const u32 n = (infoA >> 8) & 3u;
    if (n) { patch_escape(hb, infoA >> 10); if (n > 1) patch_escape(hb, infoB); }
#pragma unroll
    for (int j = 0; j < 8; ++j) { d[2 * j] = perm(hb[j], L[j], 0x05010400u); d[2 * j + 1] = perm(hb[j], L[j], 0x07030602u); }
}
__device__ __forceinline__ void load_chunk(const char* base, u32 infoA, u32 infoB, int lane, u32 (&d)[16]) {
    u32x4 a, b, c, e;
    if (infoA & 0x80u) {
        ld4_sys(a, b, c, e, base, (u32)lane * 16u);
        d[0] = a.x; d[1] = a.y; d[2] = a.z; d[3] = a.w; d[4] = b.x; d[5] = b.y; d[6] = b.z; d[7] = b.w;
        d[8] = c.x; d[9] = c.y; d[10] = c.z; d[11] = c.w; d[12] = e.x; d[13] = e.y; d[14] = e.z; d[15] = e.w;
    } else {
        ld3_sys(a, b, c, base, (u32)lane * 16u);
        const u32 L[8] = {a.x, a.y, a.z, a.w, b.x, b.y, b.z, b.w};
        const u32 N[4] = {c.x, c.y, c.z, c.w};
        decode(L, N, infoA, infoB, d);
    }
}

struct InItem { int isx, s, blk, j; };
__device__ __forceinline__ InItem in_item(int pos) {
    InItem it;
    if (pos < 384) {
        it.s = pos / 96; const int r = pos - it.s * 96;
        if (r < 32) { it.isx = 0; it.blk = r >> 2; it.j = r & 3; }
        else { it.isx = 1; it.blk = (r - 32) >> 2; it.j = r & 3; }
    } else { const int q = pos - 384; it.s = q >> 5; it.isx = 0; it.blk = 8 + ((q & 31) >> 2); it.j = q & 3; }
    return it;
}
__device__ __forceinline__ void in_addr(const Params& P, int dst, const InItem& it, int lane, const char*& sbase, u32 (&vo)[4]) {
    const int dm = (dst >> 2) & 1, dn = (dst >> 1) & 1, dk = dst & 1;
    u32 stride;
    if (it.isx) {
        const int owner = (it.blk & 1) * 4 + it.j;
        const int row = owner * 512 + dm * 256 + (it.blk >> 1) * 32;
        sbase = P.x + (size_t)row * 1024 + (size_t)(dk * 256 + it.s * 64) * 2;
        stride = 1024u;
    } else {
        const int k0 = dk * 256 + it.s * 64 + (it.j & 1) * 32, n0 = dn * 2048 + it.blk * 128 + (it.j >> 1) * 64;
        sbase = P.w + (size_t)k0 * 8192 + (size_t)n0 * 2;
        stride = 8192u;
    }
    const u32 l = ((u32)lane >> 3) * stride + ((u32)lane & 7u) * 16u;
    vo[0] = l; vo[1] = l + 8u * stride; vo[2] = l + 16u * stride; vo[3] = l + 24u * stride;
}
__device__ __forceinline__ void to_regs(const u32x4 (&r)[4], u32 (&d)[16]) {
    d[0] = r[0].x; d[1] = r[0].y; d[2] = r[0].z; d[3] = r[0].w; d[4] = r[1].x; d[5] = r[1].y; d[6] = r[1].z; d[7] = r[1].w;
    d[8] = r[2].x; d[9] = r[2].y; d[10] = r[2].z; d[11] = r[2].w; d[12] = r[3].x; d[13] = r[3].y; d[14] = r[3].z; d[15] = r[3].w;
}
__device__ __forceinline__ void store_image(const char* ib, int isx, int lane, const u32 (&d)[16]) {
    const u32 r3 = (u32)lane >> 3, p = (u32)lane & 7u;
    const u32 s0 = isx ? (r3 >> 1) : 2u * ((r3 >> 1) & 1u);
    const u32 v0 = r3 * 128u + ((p ^ s0) << 4), v1 = v0 ^ 64u;
    st_agt<0>(ib, v0, (u32x4){d[0], d[1], d[2], d[3]}); st_agt<1024>(ib, v1, (u32x4){d[4], d[5], d[6], d[7]});
    st_agt<2048>(ib, v0, (u32x4){d[8], d[9], d[10], d[11]}); st_agt<3072>(ib, v1, (u32x4){d[12], d[13], d[14], d[15]});
}
__device__ __forceinline__ size_t img_off(const InItem& it, int t) {
    const int j = it.isx ? it.j : (((it.j >> 1) & 1) * 2 + (it.j & 1));
    return (it.isx ? OFF_IMGA : OFF_IMGB) + ((size_t)(it.blk * NS + t) * 4 + j) * 4096;
}
__device__ __forceinline__ size_t rdy_off(const InItem& it, int t) { return OFF_RDY + ((size_t)((it.isx ? 0 : 64) + it.blk * 4 + it.j) * 32 + t) * 4; }
__device__ __forceinline__ void put_hdr_sys(const char* hp, u32 call, u32 a, u32 b, int lane) {
    if (lane == 0) st_sys<0>(hp, 0u, (u32x4){call, a, call, b});
}
__device__ __forceinline__ void draw_issue(u32& t, const char* ctr) {
    u64 keep; u32 z = 0, one = 1;
    asm volatile("s_nop 4\n\ts_mov_b64 %1, exec\n\ts_mov_b64 exec, 1\n\tglobal_atomic_add %0, %2, %3, %4 sc0\n\ts_mov_b64 exec, %1"
                 : "=&v"(t), "=&s"(keep) : "v"(z), "v"(one), "s"(uni(ctr)) : "memory");
}
__device__ __forceinline__ void pusher(const Params& P, Ctx& c, int cw, int wv, int lane, LAS u32x4* slots) {
    const int j = cw % 7 + 1, dst = (P.rank + j) & 7, idx = (cw / 7) * PW + wv;
    char* myws = P.ws[P.rank];
    char* dws = P.ws[dst];
    const char* din = dws + OFF_IN + (size_t)P.rank * NPOS_IN * 4096;
    const char* dhin = dws + OFF_HIN + (size_t)P.rank * NPOS_IN * 16;
    const char* dp = dws + OFF_P + (size_t)P.rank * NPOS_OUT * 4096;
    const char* dhp = dws + OFF_HP + (size_t)P.rank * NPOS_OUT * 16;
    const char* q = myws + OFF_Q + (size_t)dst * NPOS_OUT * 4096;
    const char* ctr = myws + OFF_CTL + 256 + dst * 128;
    const u32 lo = (u32)lane * 16u;
    constexpr int NPOS = NPOS_IN + (DIRECT_B ? NPOS_OUT / 2 : NPOS_OUT);
    const u32 tbase = (c.call - 1u) * (u32)NPOS - 2u * PUSH_PER_DST;
    const u32 slot0 = (u32)(size_t)slots;

    u32x4 A[4], B[4];
    u32 hA = 0, hB = 0;
    int pos = idx, nxt = idx + PUSH_PER_DST;
    {
        const InItem it = in_item(pos); const char* sb; u32 vo[4];
        in_addr(P, dst, it, lane, sb, vo);
        ld4_issue(A, sb, vo[0], vo[1], vo[2], vo[3]);
        WAITV(0); PIN4(A);
    }
    const char* prev_h = nullptr; u32 prevA = 0, prevB = 0; bool have_prev = false;
    bool stamped = false, stamped3 = false;
    for (;;) {
        const bool have_next = nxt < NPOS;
        bool ready = have_next;
        u32 nA = 0, nB = 0;
        if (have_next && nxt >= NPOS_IN) {
            const u32x4 s = lds_poll16(slot0 + (u32)(nxt - NPOS_IN) * 16u);
            const u32 s0 = rfl(s.x), s1 = rfl(s.z);
            ready = (s0 >= c.call && s1 >= c.call);
            nA = rfl(s.y); nB = rfl(s.w);
        }
        u32 tk;
        draw_issue(tk, ctr);
        {
            const char* sb = q; u32 vo[4] = {lo, lo + 1024u, lo + 2048u, lo + 3072u};
            if (ready) {
                if (nxt < NPOS_IN) in_addr(P, dst, in_item(nxt), lane, sb, vo);
                else sb = q + (size_t)(nxt - NPOS_IN) * 4096;
            }
            ld4_issue(B, sb, vo[0], vo[1], vo[2], vo[3]);
        }
        const char* db; const char* hp; u32 ia, ib;
        if (pos < NPOS_IN) {
            u32 d[16], L[8], N[4];
            to_regs(A, d);
            db = din + (size_t)pos * 4096; hp = dhin + (size_t)pos * 16;
            if (encode(d, L, N, ia, ib)) {
                st_sys<0>(db, lo, (u32x4){L[0], L[1], L[2], L[3]});
                st_sys<1024>(db, lo, (u32x4){L[4], L[5], L[6], L[7]});
                st_sys<2048>(db, lo, (u32x4){N[0], N[1], N[2], N[3]});
            } else {
                st_sys<0>(db, lo, (u32x4){d[0], d[1], d[2], d[3]});
                st_sys<1024>(db, lo, (u32x4){d[4], d[5], d[6], d[7]});
                st_sys<2048>(db, lo, (u32x4){d[8], d[9], d[10], d[11]});
                st_sys<3072>(db, lo, (u32x4){d[12], d[13], d[14], d[15]});
            }
        } else {
            if (!stamped) { STAMP(1); stamped = true; }
            db = dp + (size_t)(pos - NPOS_IN) * 4096; hp = dhp + (size_t)(pos - NPOS_IN) * 16;
            ia = hA; ib = hB;
            st_sys<0>(db, lo, A[0]); st_sys<1024>(db, lo, A[1]); st_sys<2048>(db, lo, A[2]);
            if (ia & 0x80u) st_sys<3072>(db, lo, A[3]);
        }
        int nn;
        if (ready) {
            WAITV(7);
            if (have_prev) { put_hdr_sys(prev_h, c.call, prevA, prevB, lane); WAITV(4); }
            else { WAITV(3); }
            PIN4(B); asm volatile("" : "+v"(tk));
            nn = (int)(rfl(tk) - tbase);
            A[0] = B[0]; A[1] = B[1]; A[2] = B[2]; A[3] = B[3];
            hA = nA; hB = nB;
            prev_h = hp; prevA = ia; prevB = ib; have_prev = true;
        } else {
            WAITV(0); PIN4(B); asm volatile("" : "+v"(tk));
            nn = (int)(rfl(tk) - tbase);
            if (have_prev) put_hdr_sys(prev_h, c.call, prevA, prevB, lane);
            put_hdr_sys(hp, c.call, ia, ib, lane);
            have_prev = false;
            if (!have_next) break;
            const u64 t0 = rt(); u32 polls = 0; bool ok = false;
            while (!c.dead) {
                const u32x4 s = lds_poll16(slot0 + (u32)(nxt - NPOS_IN) * 16u);
                if (rfl(s.x) >= c.call && rfl(s.z) >= c.call) { hA = rfl(s.y); hB = rfl(s.w); ok = true; break; }
                if (give_up(c, t0, polls)) break;
            }
            if (!ok) break;
            ld4_issue(A, q + (size_t)(nxt - NPOS_IN) * 4096, lo, lo + 1024u, lo + 2048u, lo + 3072u);
            WAITV(0); PIN4(A);
        }
        pos = nxt; nxt = nn;
    }
    STAMP(2);
}
__device__ __forceinline__ void scout(const Params& P, Ctx& c, int cw, int lane, LAS u32x4* slots) {
    const int j = cw % 7 + 1, dst = (P.rank + j) & 7;
    const char* hq = P.ws[P.rank] + OFF_HQ + (size_t)dst * NPOS_OUT * 16;
    const u32 vo = (u32)lane * 16u;
    u32 done = DIRECT_B ? 12u : 0u;
    const u64 t0 = rt(); u32 polls = 0;
    for (;;) {
        u32x4 h0, h1, h2, h3;
        asm volatile("s_nop 4\n\tglobal_load_dwordx4 %0, %4, %5 sc1\n\tglobal_load_dwordx4 %1, %4, %5 offset:1024 sc1\n\t"
                     "global_load_dwordx4 %2, %4, %5 offset:2048 sc1\n\tglobal_load_dwordx4 %3, %4, %5 offset:3072 sc1\n\ts_waitcnt vmcnt(0)"
                     : "=&v"(h0), "=&v"(h1), "=&v"(h2), "=&v"(h3) : "v"(vo), "s"(uni(hq)) : "memory");
        if (!(done & 1u) && h0.x >= c.call && h0.z >= c.call) { slots[lane] = h0; done |= 1u; }
        if (!(done & 2u) && h1.x >= c.call && h1.z >= c.call) { slots[64 + lane] = h1; done |= 2u; }
        if (!(done & 4u) && h2.x >= c.call && h2.z >= c.call) { slots[128 + lane] = h2; done |= 4u; }
        if (!(done & 8u) && h3.x >= c.call && h3.z >= c.call) { slots[192 + lane] = h3; done |= 8u; }
        if (__all(done == 15u)) break;
        if (!__any((done & 3u) != 0u)) __builtin_amdgcn_s_sleep(32); else __builtin_amdgcn_s_sleep(2);
        if (give_up(c, t0, polls)) break;
    }
}
__device__ __forceinline__ void unpacker(const Params& P, Ctx& c, int u, int lane) {
    char* myws = P.ws[P.rank];
    const char* pend = nullptr;
    for (int i = u; i < NPOS_IN; i += NUNP) {
        const InItem it = in_item(i); const char* sb; u32 vv[4]; u32x4 r[4]; u32 d[16];
        in_addr(P, P.rank, it, lane, sb, vv);
        ld4_issue(r, sb, vv[0], vv[1], vv[2], vv[3]);
        WAITV(0); PIN4(r);
        if (pend && lane == 0) __hip_atomic_store((GAS u32*)pend, c.call, __ATOMIC_RELAXED, __HIP_MEMORY_SCOPE_AGENT);
        to_regs(r, d);
        store_image(myws + img_off(it, it.s), it.isx, lane, d);
        pend = myws + rdy_off(it, it.s);
    }
    STAMP(1);
    const int q0 = (u + NUNP - (NPOS_IN % NUNP)) % NUNP;
    for (int q = q0; q < NPOS_IN * 7; q += NUNP) {
        const int pos = q / 7, rel = q - pos * 7 + 1;
        const int src = (P.rank + rel) & 7;
        const char* hp = myws + OFF_HIN + ((size_t)src * NPOS_IN + pos) * 16;
        u32 ia = 0, ib2 = 0; bool ok = false;
        if (!c.dead) {
            const u64 t0 = rt(); u32 polls = 0;
            for (;;) {
                const u32x4 h = poll16(hp);
                if (pend) { if (lane == 0) __hip_atomic_store((GAS u32*)pend, c.call, __ATOMIC_RELAXED, __HIP_MEMORY_SCOPE_AGENT); pend = nullptr; }
                if (rfl(h.x) >= c.call && rfl(h.z) >= c.call) { ia = rfl(h.y); ib2 = rfl(h.w); ok = true; break; }
                if (give_up(c, t0, polls)) break;
            }
        }
        if (!ok) break;
        const InItem it = in_item(pos);
        const int t = 4 + it.s * 7 + (rel - 1);
        u32 d[16];
        load_chunk(myws + OFF_IN + ((size_t)src * NPOS_IN + pos) * 4096, ia, ib2, lane, d);
        store_image(myws + img_off(it, t), it.isx, lane, d);
        pend = myws + rdy_off(it, t);
    }
    WAITV(0);
    if (pend && !c.dead && lane == 0) __hip_atomic_store((GAS u32*)pend, c.call, __ATOMIC_RELAXED, __HIP_MEMORY_SCOPE_AGENT);
    STAMP(2);
}

struct Frags { s16x8 a[2][2]; s16x4 bl[4][2], bh[4][2]; };
template <int OFF> __device__ __forceinline__ void dsr128(s16x8& r, u32 addr) { asm volatile("ds_read_b128 %0, %1 offset:%2" : "=v"(r) : "v"(addr), "i"(OFF)); }
template <int OFF> __device__ __forceinline__ void dsrtr(s16x4& r, u32 addr) { asm volatile("ds_read_b64_tr_b16 %0, %1 offset:%2" : "=v"(r) : "v"(addr), "i"(OFF)); }
__device__ __forceinline__ void mf(f32x4& acc, const s16x4& bl, const s16x4& bh, const s16x8& a) {
    s16x8 b = __builtin_shufflevector(bl, bh, 0, 1, 2, 3, 4, 5, 6, 7);
    asm volatile("v_mfma_f32_16x16x32_bf16 %0, %1, %2, %0" : "+v"(acc) : "v"(b), "v"(a));
}
template <int OFF> __device__ __forceinline__ void dma(const char* sbase, u32 voff, LAS char* dst) {
    __builtin_amdgcn_global_load_lds((const GAS void*)(sbase + voff), (LAS void*)dst, 16, OFF, 16  );
}
__device__ __forceinline__ void wait_frags(Frags& f) {
    asm volatile("s_waitcnt lgkmcnt(0)" : "+v"(f.a[0][0]), "+v"(f.a[0][1]), "+v"(f.a[1][0]), "+v"(f.a[1][1]),
        "+v"(f.bl[0][0]), "+v"(f.bl[1][0]), "+v"(f.bl[2][0]), "+v"(f.bl[3][0]), "+v"(f.bl[0][1]), "+v"(f.bl[1][1]), "+v"(f.bl[2][1]), "+v"(f.bl[3][1]) :: "memory");
    asm volatile("" : "+v"(f.bh[0][0]), "+v"(f.bh[1][0]), "+v"(f.bh[2][0]), "+v"(f.bh[3][0]), "+v"(f.bh[0][1]), "+v"(f.bh[1][1]), "+v"(f.bh[2][1]), "+v"(f.bh[3][1]));
}
struct FAddr { u32 a[2], b[4]; };
#define RA(mi, kk) dsr128<mi * 2048>(n.a[mi][kk], ad.a[kk] + so)
#define RB(ni, kk) do { dsrtr<kk * 4096>(n.bl[ni][kk], ad.b[ni] + so); dsrtr<kk * 4096 + 512>(n.bh[ni][kk], ad.b[ni] + so); } while (0)
__device__ __forceinline__ void read_all(Frags& n, const FAddr& ad, u32 so) {
    RA(0,0); RA(1,0); RB(0,0); RB(1,0); RB(2,0); RB(3,0); RA(0,1); RA(1,1); RB(0,1); RB(1,1); RB(2,1); RB(3,1);
}
#define MF(mi, ni, kk) mf(acc[mi][ni], c.bl[ni][kk], c.bh[ni][kk], c.a[mi][kk])
__device__ __forceinline__ void stage_addr(FAddr& ad, const FAddr& ad0, u32 so) {
    ad.a[0] = ad0.a[0] + so; ad.a[1] = ad0.a[1] + so; ad.b[0] = ad0.b[0] + so; ad.b[1] = ad0.b[1] + so; ad.b[2] = ad0.b[2] + so; ad.b[3] = ad0.b[3] + so;
    asm volatile("" : "+v"(ad.a[0]), "+v"(ad.a[1]), "+v"(ad.b[0]), "+v"(ad.b[1]), "+v"(ad.b[2]), "+v"(ad.b[3]));
}
__device__ __forceinline__ void kstep1(f32x4 (&acc)[2][4], const Frags& c, Frags& n, const FAddr& ad) {
#undef RA
#undef RB
#define RA(mi, kk) dsr128<mi * 2048>(n.a[mi][kk], ad.a[kk])
#define RB(ni, kk) do { dsrtr<kk * 4096>(n.bl[ni][kk], ad.b[ni]); dsrtr<kk * 4096 + 512>(n.bh[ni][kk], ad.b[ni]); } while (0)
    MF(0,0,0); RA(0,0); RA(1,0);
    MF(0,1,0); RB(0,0);
    MF(0,2,0); RB(1,0);
    MF(0,3,0); RB(2,0);
    MF(1,0,0); RB(3,0);
    MF(1,1,0); RA(0,1); RA(1,1);
    MF(1,2,0); RB(0,1);
    MF(1,3,0); RB(1,1);
    MF(0,0,1); RB(2,1);
    MF(0,1,1); RB(3,1);
}
#define GR_PF 0
constexpr int PF = GR_PF;
__device__ __forceinline__ void kstep2(f32x4 (&acc)[2][4], const Frags& c, const char* sb, u32 voff, LAS char* dst, const char* pfb, u32 pfo, u32& sink, FAddr& adn, const FAddr& ad0, u32 son) {
    if (PF) asm volatile("s_nop 4\n\tglobal_load_dword %0, %1, %2 sc1" : "+v"(sink) : "v"(pfo), "s"(uni(pfb)) : "memory");
    MF(0,2,1); dma<0>(sb, voff, dst);
    MF(0,3,1); dma<1024>(sb, voff, dst);
    MF(1,0,1); dma<2048>(sb, voff, dst);
    MF(1,1,1); dma<3072>(sb, voff, dst);
    stage_addr(adn, ad0, son);
    MF(1,2,1);
    MF(1,3,1);
}
#undef MF
#undef RA
#undef RB
__device__ __noinline__ u32 wait_avail(const char* tmo, u32 call, const char* rdy0, const char* rdy1, u32 u) {
    Ctx c; c.tmo = tmo; c.call = call; c.dead = false;
    const char* line = u < NS ? rdy0 : rdy1;
    const u32 base = u < NS ? 0u : (u32)NS;
    const u64 t0 = rt(); u32 polls = 0;
    for (;;) {
        u32 v; const u32 vo = (threadIdx.x & 31u) * 4u;
        asm volatile("s_nop 4\n\tglobal_load_dword %0, %1, %2 sc1\n\ts_waitcnt vmcnt(0)" : "=&v"(v) : "v"(vo), "s"(uni(line)) : "memory");
        const u32 miss = ~(u32)__ballot(v >= call);
        const u32 av = base + (miss ? (u32)__builtin_ctz(miss) : (u32)NS);
        if (av > u) return av;
        if (give_up(c, t0, polls)) return 0x80000000u;
    }
}
__device__ __forceinline__ u32 cvt_pk_bf16(float lo, float hi) { u32 r; asm volatile("v_cvt_pk_bf16_f32 %0, %1, %2" : "=v"(r) : "v"(lo), "v"(hi)); return r; }
template <int N> __device__ __forceinline__ void waitv() { asm volatile("s_waitcnt vmcnt(%0)" :: "i"(N) : "memory"); }

__device__ __forceinline__ float bf_lo(u32 v) { return __uint_as_float(v << 16); }
__device__ __forceinline__ float bf_hi(u32 v) { return __uint_as_float(v & 0xFFFF0000u); }

#define GR_TRACE(u) do { } while (0)
__device__ __forceinline__ void gemm_stream(const Params& P, Ctx& c, LAS char* lds, int rb, int cb0, int ntile, int w, int lane) {
    char* myws = P.ws[P.rank];
    const int wm = w & 3, wn = w >> 2;
    const u32 ldsbase = (u32)(size_t)lds;
    const char* src0 = uni(myws + (w < 4 ? OFF_IMGA + ((size_t)rb * NS * 4 + w) * 4096 : OFF_IMGB + ((size_t)cb0 * NS * 4 + (w - 4)) * 4096));
    const char* src1 = (w < 4 || ntile == 1) ? src0 : src0 + (size_t)8 * NS * 4 * 4096;
    const char* rdy0 = myws + OFF_RDY + (size_t)(w < 4 ? rb * 4 + w : 64 + cb0 * 4 + (w - 4)) * 128;
    const char* rdy1 = (w < 4 || ntile == 1) ? rdy0 : rdy0 + 8 * 4 * 128;
    const u32 ulast = (u32)(ntile * NS - 1);
    const u32 voff = (u32)lane * 16u;
    LAS char* ldsw = lds + w * 4096;
    FAddr ad;
    {
        const u32 r = (u32)lane & 15u, g = (u32)lane >> 4;
        ad.a[0] = ldsbase + wm * 4096 + r * 128u + ((g ^ (r >> 1)) << 4);
        ad.a[1] = ad.a[0] ^ 64u;
        const u32 q = r >> 2, pp = r & 3u, sw = (q >> 1) + 2u * (g & 1u);
#pragma unroll
        for (int ni = 0; ni < 4; ++ni) ad.b[ni] = ldsbase + 16384 + wn * 8192 + (8u * g + q) * 128u + (((u32)ni ^ sw) << 5) + pp * 8u;
    }
    u32 avail = 0;
    const u32 pfo = ((u32)lane & 31u) * 128u; u32 sink = 0;
    f32x4 acc[2][4];
#pragma unroll
    for (int mi = 0; mi < 2; ++mi)
#pragma unroll
        for (int ni = 0; ni < 4; ++ni) acc[mi][ni] = (f32x4){0.f, 0.f, 0.f, 0.f};
    Frags f0, f1;
    if (avail < NST) avail = rfl(wait_avail(c.tmo, c.call, rdy0, rdy1, NST - 1));
#pragma unroll
    for (int s = 0; s < NST; ++s) {
        LAS char* d = ldsw + s * 32768; const char* g = src0 + (size_t)s * 16384;
        if (PF) asm volatile("s_nop 4\n\tglobal_load_dword %0, %1, %2 sc1" : "+v"(sink) : "v"(pfo), "s"(uni(g)) : "memory");
        dma<0>(g, voff, d); dma<1024>(g, voff, d); dma<2048>(g, voff, d); dma<3072>(g, voff, d);
    }
    waitv<(PF ? 5 : 4) * (NST - 1)>(); BAR();
    read_all(f0, ad, 0u);
    wait_frags(f0);
    asm volatile("s_nop 1");
    u32 bd = 0, br = 32768;
    const char* pend_h = nullptr; u32 pendA = 0, pendB = 0;
    constexpr u32 PUBU = NS + NST - 2;
    FAddr adr; stage_addr(adr, ad, br);
    for (int tile = 0; tile < ntile; ++tile) {
        const int cb = cb0 + 8 * tile;
        for (int s = 0; s < NS; s += 2) {
#define GR_STEP(S, CUR, NXT) do { \
            const u32 u = (u32)(tile * NS + (S)), un = u + NST;         \
            waitv<(PF ? 5 : 4) * (NST - 2)>(); BAR(); SCHED(); \
            GR_TRACE(u); \
            kstep1(acc, CUR, NXT, adr); \
            const char* sn = ((un & NS) ? src1 : src0) + (size_t)(un & (NS - 1)) * 16384; \
            const u32 brn = br + 32768 == NST * 32768 ? 0 : br + 32768; \
            if (min(un, ulast) >= avail) avail = rfl(wait_avail(c.tmo, c.call, rdy0, rdy1, min(un, ulast))); \
            if (u == PUBU) { if (avail != 0x80000000u && lane == 0) st_sys<0>(pend_h, 0u, (u32x4){c.call, pendA, c.call, pendB}); } \
            { const u32 up = un + PF; const bool okp = PF && up <= ulast && up < avail; \
              kstep2(acc, CUR, sn, voff, ldsw + bd, okp ? ((up & NS) ? src1 : src0) + (size_t)(up & (NS - 1)) * 16384 : sn, pfo, sink, adr, ad, brn); } \
            bd = br; br = brn; \
            wait_frags(NXT); SCHED(); } while (0)
            GR_STEP(s, f0, f1);
            GR_STEP(s + 1, f1, f0);
#undef GR_STEP
        }
        asm volatile("s_nop 15\n\ts_nop 15" ::: "memory");
        asm volatile("" : "+v"(acc[0][0]), "+v"(acc[0][1]), "+v"(acc[0][2]), "+v"(acc[0][3]), "+v"(acc[1][0]), "+v"(acc[1][1]), "+v"(acc[1][2]), "+v"(acc[1][3]));
        u32 d[16], L[8], N[4], ia, ib;
#pragma unroll
        for (int mi = 0; mi < 2; ++mi)
#pragma unroll
            for (int ni = 0; ni < 4; ++ni) {
                d[mi * 8 + ni * 2] = cvt_pk_bf16(acc[mi][ni][0], acc[mi][ni][1]);
                d[mi * 8 + ni * 2 + 1] = cvt_pk_bf16(acc[mi][ni][2], acc[mi][ni][3]);
                acc[mi][ni] = (f32x4){0.f, 0.f, 0.f, 0.f};
            }
        const int owner = (rb & 1) * 4 + wm;
        const int cidx = (cb >> 3) * 128 + (rb >> 1) * 16 + (cb & 7) * 2 + wn;
        const bool mine = owner == P.rank;
        const u32 lo = (u32)lane * 16u;
        const bool direct = DIRECT_B && cb >= 8;
        char* tws = (direct || mine) ? P.ws[owner] : myws;
        const int slot = ((direct || mine) ? P.rank : owner) * NPOS_OUT + cidx;
        const char* db = tws + ((direct || mine) ? OFF_P : OFF_Q) + (size_t)slot * 4096;
        const char* hp = tws + ((direct || mine) ? OFF_HP : OFF_HQ) + (size_t)slot * 16;
        const bool packed = encode(d, L, N, ia, ib);
        st_sys<0>(db, lo, packed ? (u32x4){L[0], L[1], L[2], L[3]} : (u32x4){d[0], d[1], d[2], d[3]});
        st_sys<1024>(db, lo, packed ? (u32x4){L[4], L[5], L[6], L[7]} : (u32x4){d[4], d[5], d[6], d[7]});
        st_sys<2048>(db, lo, packed ? (u32x4){N[0], N[1], N[2], N[3]} : (u32x4){d[8], d[9], d[10], d[11]});
        if (!packed) st_sys<3072>(db, lo, (u32x4){d[12], d[13], d[14], d[15]});
        pend_h = hp; pendA = ia; pendB = ib;
    }
    if (PF) asm volatile("s_waitcnt vmcnt(0)" : "+v"(sink) :: "memory");
    if (avail == 0x80000000u) c.dead = true;
    WAITV(0);
    if (!c.dead && lane == 0) st_sys<0>(pend_h, 0u, (u32x4){c.call, pendA, c.call, pendB});
    BAR();
}

__device__ __forceinline__ void unpack_regs(const u32x4& a, const u32x4& b, const u32x4& cc, const u32x4& e, u32 infoA, u32 infoB, int lane, u32 (&d)[16]) {
    if (infoA & 0x80u) {
        d[0] = a.x; d[1] = a.y; d[2] = a.z; d[3] = a.w; d[4] = b.x; d[5] = b.y; d[6] = b.z; d[7] = b.w;
        d[8] = cc.x; d[9] = cc.y; d[10] = cc.z; d[11] = cc.w; d[12] = e.x; d[13] = e.y; d[14] = e.z; d[15] = e.w;
    } else {
        const u32 L[8] = {a.x, a.y, a.z, a.w, b.x, b.y, b.z, b.w};
        const u32 N[4] = {cc.x, cc.y, cc.z, cc.w};
        decode(L, N, infoA, infoB, d);
    }
}

__device__ __forceinline__ void gemm_role(const Params& P, Ctx& c, LAS char* lds, int g, int w, int lane) {
    char* myws = P.ws[P.rank];
    const int xcd = g & 7;
    STAMP(0);
    {
        int rb, cb0, ntile;
        if (g >= NRED) {
            const int i = (g - NRED) >> 3;
            rb = 8 + (xcd & 3) * 2 + (i & 1); cb0 = 8 + (xcd >> 2) * 4 + (i >> 1); ntile = 1;
        } else {
            const int i = g >> 3;
            rb = (xcd & 3) * 4 + (i & 3); cb0 = (xcd >> 2) * 4 + (i >> 2); ntile = rb < 8 ? 2 : 1;
        }
        gemm_stream(P, c, lds, rb, cb0, ntile, w, lane);
        if (g >= NRED) return;
    }
    const int quad = w >> 1, cidx = (w & 1) * 128 + g;
    const int qm = quad >> 1, qn = quad & 1;
    const int s0 = (qm << 2) | (qn << 1);
    float y[32];
    {
        u32 a0 = 0, b0 = 0, a1 = 0, b1 = 0; bool ok = false;
        if (!c.dead) {
            const char* hb = myws + OFF_HP + ((size_t)s0 * NPOS_OUT + cidx) * 16;
            const u32 vo = (u32)(lane >> 5) * (NPOS_OUT * 16u);
            const u64 t0 = rt(); u32 polls = 0;
            for (;;) {
                u32x4 h;
                asm volatile("s_nop 4\n\tglobal_load_dwordx4 %0, %1, %2 sc0 sc1\n\ts_waitcnt vmcnt(0)" : "=&v"(h) : "v"(vo), "s"(uni(hb)) : "memory");
                if (__all(h.x >= c.call && h.z >= c.call)) {
                    a0 = (u32)__builtin_amdgcn_readlane((int)h.y, 0); b0 = (u32)__builtin_amdgcn_readlane((int)h.w, 0);
                    a1 = (u32)__builtin_amdgcn_readlane((int)h.y, 32); b1 = (u32)__builtin_amdgcn_readlane((int)h.w, 32);
                    ok = true; break;
                }
                if (give_up(c, t0, polls)) break;
            }
        }
        u32 d0[16], d1[16];
        if (ok) {
            u32x4 p0, p1, p2, p3, q0, q1, q2, q3;
            const char* pb = myws + OFF_P + ((size_t)s0 * NPOS_OUT + cidx) * 4096;
            const u32 v0 = (u32)lane * 16u, v1 = v0 + NPOS_OUT * 4096u;
            asm volatile("s_nop 4\n\tglobal_load_dwordx4 %0, %8, %10 sc0 sc1\n\tglobal_load_dwordx4 %1, %8, %10 offset:1024 sc0 sc1\n\t"
                         "global_load_dwordx4 %2, %8, %10 offset:2048 sc0 sc1\n\tglobal_load_dwordx4 %4, %9, %10 sc0 sc1\n\t"
                         "global_load_dwordx4 %5, %9, %10 offset:1024 sc0 sc1\n\tglobal_load_dwordx4 %6, %9, %10 offset:2048 sc0 sc1\n\t"
                         "global_load_dwordx4 %3, %8, %10 offset:3072 sc0 sc1\n\tglobal_load_dwordx4 %7, %9, %10 offset:3072 sc0 sc1\n\ts_waitcnt vmcnt(0)"
                         : "=&v"(p0), "=&v"(p1), "=&v"(p2), "=&v"(p3), "=&v"(q0), "=&v"(q1), "=&v"(q2), "=&v"(q3)
                         : "v"(v0), "v"(v1), "s"(uni(pb)) : "memory");
            unpack_regs(p0, p1, p2, p3, a0, b0, lane, d0);
            unpack_regs(q0, q1, q2, q3, a1, b1, lane, d1);
        } else {
#pragma unroll
            for (int k = 0; k < 16; ++k) { d0[k] = 0; d1[k] = 0; }
        }
#pragma unroll
        for (int k = 0; k < 16; ++k) { y[2 * k] = bf_lo(d0[k]) + bf_lo(d1[k]); y[2 * k + 1] = bf_hi(d0[k]) + bf_hi(d1[k]); }
    }
    STAMP(3);
    u32 mx = 0;
#pragma unroll
    for (int k = 0; k < 32; ++k) mx = max(mx, __float_as_uint(y[k]) & 0x7FFFFFFFu);
    mx = wave_umax(mx);
    LAS u32* red = (LAS u32*)lds;
    if (lane == 0) red[w] = mx;
    __syncthreads();
    if (w == 0 && lane < NR) {
        u32 m = 0;
#pragma unroll
        for (int k = 0; k < 8; ++k) m = max(m, red[k]);
        if (!c.dead)
            __hip_atomic_store((GAS u64*)(P.ws[lane] + OFF_AMAX + ((size_t)P.rank * NRED + g) * 8), ((u64)m << 32) | c.call, __ATOMIC_RELAXED, __HIP_MEMORY_SCOPE_SYSTEM);
    }
    u32 gm = 0;
    if (!c.dead) {
        const u64 t0 = rt(); u32 polls = 0;
        const char* ab = myws + OFF_AMAX; const u32 vo = (u32)(w * 64 + lane) * 16u;
        for (;;) {
            u32x4 h;
            asm volatile("s_nop 4\n\tglobal_load_dwordx4 %0, %1, %2 sc0 sc1\n\ts_waitcnt vmcnt(0)" : "=&v"(h) : "v"(vo), "s"(uni(ab)) : "memory");
            if (__all(h.x >= c.call && h.z >= c.call)) { gm = max(h.y, h.w); break; }
            if (give_up(c, t0, polls)) break;
        }
    }
    gm = wave_umax(gm);
    if (lane == 0) red[8 + w] = gm;
    __syncthreads();
    u32 am = 0;
#pragma unroll
    for (int k = 0; k < 8; ++k) am = max(am, red[8 + k]);
    STAMP(4);
    const u32 tm = __hip_atomic_load((const GAS u32*)c.tmo, __ATOMIC_RELAXED, __HIP_MEMORY_SCOPE_AGENT);
    const bool bad = c.dead || tm == c.call;
    const float scale = __uint_as_float(am) / 448.0f;
    const float inv = 1.0f / scale;
    const int ph = cidx >> 7, rbh = (cidx >> 4) & 7, cb7 = (cidx >> 1) & 7, wn = cidx & 1;
    char* orow = P.out + (size_t)(qm * 256 + rbh * 32 + (lane & 15)) * 4096 + qn * 2048 + (ph * 8 + cb7) * 128 + wn * 64 + (lane >> 4) * 16;
#pragma unroll
    for (int mi = 0; mi < 2; ++mi) {
        u32x4 o;
#pragma unroll
        for (int k = 0; k < 4; ++k) {
            const float* v = y + mi * 16 + k * 4;
            int p = __builtin_amdgcn_cvt_pk_fp8_f32(v[0] * inv, v[1] * inv, 0, false);
            p = __builtin_amdgcn_cvt_pk_fp8_f32(v[2] * inv, v[3] * inv, p, true);
            o[k] = bad ? 0x7F7F7F7Fu : (u32)p;
        }
        auto s0 = __builtin_amdgcn_permlane32_swap(o[0], o[2], false, false);
        auto s1 = __builtin_amdgcn_permlane32_swap(o[1], o[3], false, false);
        auto t0 = __builtin_amdgcn_permlane16_swap(s0[0], s1[0], false, false);
        auto t1 = __builtin_amdgcn_permlane16_swap(s0[1], s1[1], false, false);
        *(u32x4*)(orow + (size_t)mi * 16 * 4096) = (u32x4){t0[0], t0[1], t1[0], t1[1]};
    }
    if (g == 0 && w == 0 && lane == 0) *(u32*)(myws + OFF_CTL) = c.call;
    STAMP(5);
}

__global__ void __launch_bounds__(512, 2) gemm_rs_kernel(Params P) {
    extern __shared__ __attribute__((aligned(16))) char smem[];
    LAS char* lds = (LAS char*)smem;
    const int tid = threadIdx.x, w = (int)rfl((u32)tid >> 6), lane = tid & 63;
    char* myws = P.ws[P.rank];
    Ctx c;
    c.tmo = myws + OFF_CTL + 128;
    c.call = rfl(__hip_atomic_load((const GAS u32*)(myws + OFF_CTL), __ATOMIC_RELAXED, __HIP_MEMORY_SCOPE_AGENT)) + 1u;
    c.dead = false;
    const int b = blockIdx.x;
    if (b < NGEMM) { gemm_role(P, c, lds, b, w, lane); return; }
    const int cw = b - NGEMM;
    STAMP(0);
    if (cw < NPUSHWG) {
        LAS u32x4* slots = (LAS u32x4*)lds;
        if (tid < NPOS_OUT) slots[tid] = (u32x4){0u, 0u, 0u, 0u};
        __syncthreads();
        if (w < PW) pusher(P, c, cw, w, lane, slots);
        else if (w == PW) scout(P, c, cw, lane, slots);
    } else {
        unpacker(P, c, (cw - NPUSHWG) * 8 + w, lane);
    }
}

}

extern "C" void kernel_launch(int rank, int nranks, ncclComm_t comm, hipStream_t stream,
                              void* const* d_in, const int* in_sizes, int n_in,
                              void* d_out, int out_size,
                              const int* in_dtypes, int out_dtype,
                              void* const* d_out_all,
                              void* const* d_ws_all, size_t ws_bytes) {
    (void)comm; (void)in_dtypes; (void)out_dtype; (void)d_out_all;
    if (nranks != gr::NR || (unsigned)rank >= (unsigned)gr::NR || n_in < 2 || ws_bytes < gr::WS_NEED ||
        in_sizes[0] != 4096 * 512 || in_sizes[1] != 512 * 4096 || out_size != 512 * 4096) {
        fprintf(stderr, "kernel_launch: unexpected shapes\n");
        return;
    }
    static bool attr_set = false;
    if (!attr_set) {
        hipError_t e = hipFuncSetAttribute((const void*)gr::gemm_rs_kernel, hipFuncAttributeMaxDynamicSharedMemorySize, gr::LDS_BYTES);
        if (e != hipSuccess) { fprintf(stderr, "hipFuncSetAttribute: %s\n", hipGetErrorString(e)); return; }
        attr_set = true;
    }
    gr::Params p;
    p.x = (const char*)d_in[0]; p.w = (const char*)d_in[1]; p.out = (char*)d_out;
    for (int i = 0; i < gr::NR; ++i) p.ws[i] = (char*)d_ws_all[i];
    p.rank = rank; p.pad = 0;
    gr::gemm_rs_kernel<<<dim3(256), dim3(512), gr::LDS_BYTES, stream>>>(p);
    hipError_t e = hipGetLastError();
    if (e != hipSuccess) fprintf(stderr, "launch: %s\n", hipGetErrorString(e));
}
```

```cpp
#include <hip/hip_runtime.h>
#include <rccl/rccl.h>
#include <cstdio>
#include <cstdint>

#define LAS __attribute__((address_space(3)))
#define GAS __attribute__((address_space(1)))

namespace gr {

typedef unsigned u32;
typedef unsigned long long u64;
typedef unsigned u32x4 __attribute__((ext_vector_type(4)));
typedef unsigned short u16x2 __attribute__((ext_vector_type(2)));
typedef short s16x4 __attribute__((ext_vector_type(4)));
typedef short s16x8 __attribute__((ext_vector_type(8)));
typedef float f32x4 __attribute__((ext_vector_type(4)));

constexpr int NR = 8;
constexpr int NS = 32;
constexpr int NRED = 128;
constexpr int NGEMM = 192;
#define GR_PW 3
#define GR_PWG 5
constexpr int NCOMM = 256 - NGEMM;
constexpr int PWG = GR_PWG;
constexpr int NPUSHWG = 7 * PWG;
constexpr int PW = GR_PW;
static_assert(PW < 12 && NPUSHWG < NCOMM, "wave roles");
constexpr int PUSH_PER_DST = PWG * PW;
constexpr int NQUAD = (NCOMM - NPUSHWG) * 3;
constexpr int NTHR = 768;
#define GR_DIRECT 1
constexpr bool DIRECT_B = GR_DIRECT;
constexpr int NPOS_IN = 512;
constexpr int NPOS_OUT = 256;
#define GR_STAGED 64
constexpr int NSTAGED = GR_STAGED;
constexpr u64 DEADLINE = 40000000ull;

constexpr size_t OFF_CTL = 0;
#define GR_AMAXR 4
constexpr int AMAXR = GR_AMAXR;
constexpr size_t OFF_LOG = 16384;
constexpr size_t OFF_FLG = OFF_LOG + 4096;
constexpr size_t OFF_TKT = OFF_FLG + 4096;
constexpr size_t OFF_HIN = 32768;
constexpr size_t OFF_HP = OFF_HIN + 65536;
constexpr size_t OFF_HQ = OFF_HP + 32768;
constexpr size_t OFF_AMAX = OFF_HQ + 32768;
static_assert(AMAXR <= 8 && OFF_AMAX + 8 * 8192 <= 262144, "amax slots");
constexpr size_t OFF_DBG = 262144;
constexpr size_t OFF_IN = 1u << 20;
constexpr size_t OFF_IMGA = OFF_IN + (16u << 20);
constexpr size_t OFF_IMGB = OFF_IMGA + (8u << 20);
constexpr size_t OFF_P = OFF_IMGB + (8u << 20);
constexpr size_t OFF_Q = OFF_P + (8u << 20);
constexpr size_t WS_NEED = OFF_Q + (8u << 20);
#define GR_NST 5
constexpr int NST = GR_NST;
constexpr int LDS_BYTES = NST * 32768;

struct Params {
    const char* x;
    const char* w;
    char* out;
    char* ws[NR];
    int rank;
    int pad;
};

#define GR_COMMFIRST 1
__device__ __forceinline__ int bid() { return GR_COMMFIRST ? (int)((blockIdx.x + 192u) & 255u) : (int)blockIdx.x; }
#define WAITV(n) asm volatile("s_waitcnt vmcnt(" #n ")" ::: "memory")
#define BAR() __builtin_amdgcn_s_barrier()
#define SCHED() __builtin_amdgcn_sched_barrier(0)

__device__ __forceinline__ u64 rt() { return __builtin_amdgcn_s_memrealtime(); }
__device__ __forceinline__ u32 rfl(u32 v) { return (u32)__builtin_amdgcn_readfirstlane((int)v); }

#define STAMP(slot) do { } while (0)
#define STAMPV(slot, val) do { } while (0)

#define CT(arr, idx) do { } while (0)
__device__ __forceinline__ const char* uni(const char* p) {
    const u64 v = (u64)p;
    return (const char*)(((u64)rfl((u32)(v >> 32)) << 32) | (u64)rfl((u32)v));
}
template <int OFF> __device__ __forceinline__ void st_sys(const char* sbase, u32 voff, u32x4 v) {
    asm volatile("s_nop 4\n\tglobal_store_dwordx4 %0, %1, %2 offset:%3 sc0 sc1\n\ts_nop 1" :: "v"(voff), "v"(v), "s"(uni(sbase)), "i"(OFF) : "memory");
}
#define GR_STBITS "sc1"
template <int OFF> __device__ __forceinline__ void st_agt(const char* sbase, u32 voff, u32x4 v) {
    asm volatile("s_nop 4\n\tglobal_store_dwordx4 %0, %1, %2 offset:%3 " GR_STBITS "\n\ts_nop 1" :: "v"(voff), "v"(v), "s"(uni(sbase)), "i"(OFF) : "memory");
}
__device__ __forceinline__ u32x4 poll16(const char* sbase) {
    u32x4 r; u32 z = 0;
    asm volatile("s_nop 4\n\tglobal_load_dwordx4 %0, %1, %2 sc0 sc1\n\ts_waitcnt vmcnt(0)" : "=&v"(r) : "v"(z), "s"(uni(sbase)) : "memory");
    return r;
}
#define GR_LDBITS "sc0 sc1"
__device__ __forceinline__ void ld3_sys(u32x4& a, u32x4& b, u32x4& c, const char* sbase, u32 voff) {
    asm volatile("s_nop 4\n\tglobal_load_dwordx4 %0, %3, %4 " GR_LDBITS "\n\tglobal_load_dwordx4 %1, %3, %4 offset:1024 " GR_LDBITS "\n\t"
                 "global_load_dwordx4 %2, %3, %4 offset:2048 " GR_LDBITS "\n\ts_waitcnt vmcnt(0)"
                 : "=&v"(a), "=&v"(b), "=&v"(c) : "v"(voff), "s"(uni(sbase)) : "memory");
}
__device__ __forceinline__ void ld4_sys(u32x4& a, u32x4& b, u32x4& c, u32x4& d, const char* sbase, u32 voff) {
    asm volatile("s_nop 4\n\tglobal_load_dwordx4 %0, %4, %5 sc0 sc1\n\tglobal_load_dwordx4 %1, %4, %5 offset:1024 sc0 sc1\n\t"
                 "global_load_dwordx4 %2, %4, %5 offset:2048 sc0 sc1\n\tglobal_load_dwordx4 %3, %4, %5 offset:3072 sc0 sc1\n\ts_waitcnt vmcnt(0)"
                 : "=&v"(a), "=&v"(b), "=&v"(c), "=&v"(d) : "v"(voff), "s"(uni(sbase)) : "memory");
}
__device__ __forceinline__ void ld4_issue(u32x4 (&r)[4], const char* sbase, u32 v0, u32 v1, u32 v2, u32 v3) {
    asm volatile("s_nop 4\n\tglobal_load_dwordx4 %0, %4, %8 sc1\n\tglobal_load_dwordx4 %1, %5, %8 sc1\n\t"
                 "global_load_dwordx4 %2, %6, %8 sc1\n\tglobal_load_dwordx4 %3, %7, %8 sc1"
                 : "=&v"(r[0]), "=&v"(r[1]), "=&v"(r[2]), "=&v"(r[3]) : "v"(v0), "v"(v1), "v"(v2), "v"(v3), "s"(uni(sbase)) : "memory");
}
#define PIN4(r) asm volatile("" : "+v"(r[0]), "+v"(r[1]), "+v"(r[2]), "+v"(r[3]))

__device__ __forceinline__ u32x4 lds_poll16(u32 addr) {
    u32x4 r;
    asm volatile("ds_read_b128 %0, %1\n\ts_waitcnt lgkmcnt(0)" : "=&v"(r) : "v"(addr) : "memory");
    return r;
}
__device__ __forceinline__ u32 call_number() { u32 v; asm volatile("v_readfirstlane_b32 %0, v163" : "=s"(v)); return v + 1u; }
struct Ctx { const char* tmo; u32 call; bool dead;
};
__device__ __forceinline__ bool give_up(Ctx& c, u64 t0, u32& polls) {
    __builtin_amdgcn_s_sleep(2);
    ++polls;
    if ((polls & 15u) == 0u) {
        u32 t = __hip_atomic_load((const GAS u32*)c.tmo, __ATOMIC_RELAXED, __HIP_MEMORY_SCOPE_AGENT);
        if (rfl(t) == c.call) { c.dead = true; return true; }
        if (rt() - t0 > DEADLINE) {
            __hip_atomic_store((GAS u32*)c.tmo, c.call, __ATOMIC_RELAXED, __HIP_MEMORY_SCOPE_AGENT);
            c.dead = true; return true;
        }
    }
    return false;
}
__device__ __forceinline__ bool wait_hdr(Ctx& c, const char* hp, u32& a, u32& b) {
    a = 0; b = 0;
    if (c.dead) return false;
    const u64 t0 = rt(); u32 polls = 0;
    for (;;) {
        u32x4 h = poll16(hp);
        const u32 s0 = rfl(h.x), s1 = rfl(h.z);
        if (s0 >= c.call && s1 >= c.call) { a = rfl(h.y); b = rfl(h.w); return true; }
        if (give_up(c, t0, polls)) return false;
    }
}

__device__ __forceinline__ u32 perm(u32 s0, u32 s1, u32 sel) { return __builtin_amdgcn_perm(s0, s1, sel); }
__device__ __forceinline__ u32 wave_umax(u32 v) {
    v = max(v, (u32)__builtin_amdgcn_update_dpp(0, (int)v, 0x111, 0xf, 0xf, true));
    v = max(v, (u32)__builtin_amdgcn_update_dpp(0, (int)v, 0x112, 0xf, 0xf, true));
    v = max(v, (u32)__builtin_amdgcn_update_dpp(0, (int)v, 0x114, 0xf, 0xf, true));
    v = max(v, (u32)__builtin_amdgcn_update_dpp(0, (int)v, 0x118, 0xf, 0xf, true));
    v = max(v, (u32)__builtin_amdgcn_update_dpp(0, (int)v, 0x142, 0xa, 0xf, true));
    v = max(v, (u32)__builtin_amdgcn_update_dpp(0, (int)v, 0x143, 0xc, 0xf, true));
    return (u32)__builtin_amdgcn_readlane((int)v, 63);
}
__device__ __forceinline__ u32 pkmax(u32 a, u32 b) {
    return __builtin_bit_cast(u32, __builtin_elementwise_max(__builtin_bit_cast(u16x2, a), __builtin_bit_cast(u16x2, b)));
}
__device__ __forceinline__ u32 byte_of(u32 h0, u32 h1, u32 h2, u32 h3, u32 h4, u32 h5, u32 h6, u32 h7, u32 v) {
    asm volatile("" : "+v"(h0), "+v"(h1), "+v"(h2), "+v"(h3), "+v"(h4), "+v"(h5), "+v"(h6), "+v"(h7));
    const bool b0 = v & 4u, b1 = v & 8u, b2 = v & 16u;
    const u32 x0 = b0 ? h1 : h0, x1 = b0 ? h3 : h2, x2 = b0 ? h5 : h4, x3 = b0 ? h7 : h6;
    const u32 y0 = b1 ? x1 : x0, y1 = b1 ? x3 : x2;
    return ((b2 ? y1 : y0) >> ((v & 3u) * 8u)) & 0xFFu;
}
__device__ __forceinline__ void collect_escapes(const u32 (&U)[8], const u32 (&H)[8], u32 base, u32& infoA, u32& infoB) {
    u32 mk = 0;
#pragma unroll
    for (int j = 0; j < 8; ++j) mk |= ((((~U[j] >> 7) & 0x01010101u) * 0x01020408u) >> 24 & 15u) << (4 * j);
    const u32 lane = threadIdx.x & 63u, cnt = (u32)__builtin_popcount(mk);
    const u32 v1 = (u32)__builtin_ctz(mk | 0x80000000u), mk2 = mk & (mk - 1u), v2 = (u32)__builtin_ctz(mk2 | 0x80000000u);
    const u32 e1 = (v1 * 64u + lane) | (byte_of(H[0], H[1], H[2], H[3], H[4], H[5], H[6], H[7], v1) << 11), e2 = (v2 * 64u + lane) | (byte_of(H[0], H[1], H[2], H[3], H[4], H[5], H[6], H[7], v2) << 11);
    u64 any = __ballot(mk != 0u);
    const int l1 = __builtin_ctzll(any); any &= any - 1;
    const u32 c1 = (u32)__builtin_amdgcn_readlane((int)cnt, l1), a1 = (u32)__builtin_amdgcn_readlane((int)e1, l1), a2 = (u32)__builtin_amdgcn_readlane((int)e2, l1);
    u32 n = c1, f0 = a1, f1 = a2;
    if (any) {
        const int l2 = __builtin_ctzll(any); any &= any - 1;
        n += (u32)__builtin_amdgcn_readlane((int)cnt, l2);
        f1 = (u32)__builtin_amdgcn_readlane((int)e1, l2);
        if (any) n = 3;
    }
    if (n > 2) { infoA = 0x80u; infoB = 0; } else { infoA = base | (n << 8) | (f0 << 10); infoB = n > 1 ? f1 : 0u; }
}
#define GR_INACK 0
constexpr u32 XM0 = 0xA5C3965Au, XM1 = 0x5A96C3A5u, XM2 = 0x80918008u;
__device__ __forceinline__ bool arrived(const u32x4& a, const u32x4& b, const u32x4& c) { return __ballot(min(min(a.x, b.x), c.x) == 0u) == 0; }
__device__ __forceinline__ bool encode(const u32 (&d)[16], u32 (&L)[8], u32 (&N)[4], u32& infoA, u32& infoB) {
    u32 m[8];
#pragma unroll
    for (int i = 0; i < 8; ++i) m[i] = pkmax(d[2 * i] & 0x7FFF7FFFu, d[2 * i + 1] & 0x7FFF7FFFu);
    const u32 m4 = pkmax(pkmax(pkmax(m[0], m[1]), pkmax(m[2], m[3])), pkmax(pkmax(m[4], m[5]), pkmax(m[6], m[7])));
    const u32 mx = wave_umax(max(m4 & 0xFFFFu, m4 >> 16)) >> 8;
    const u32 base = mx > 7u ? mx - 7u : 0u;
    const u32 rep = base * 0x01010101u;
    u32 H[8], U[8], c[8];
#pragma unroll
    for (int j = 0; j < 8; ++j) {
        L[j] = perm(d[2 * j + 1], d[2 * j], 0x06040200u);
        H[j] = perm(d[2 * j + 1], d[2 * j], 0x07050301u);
        U[j] = (H[j] | 0x80808080u) - rep;
        c[j] = (U[j] & 0x07070707u) | ((H[j] >> 4) & 0x08080808u);
    }
    const u32 all = ((U[0] & U[1]) & (U[2] & U[3])) & ((U[4] & U[5]) & (U[6] & U[7]));
#pragma unroll
    for (int j = 0; j < 4; ++j) N[j] = c[2 * j] | (c[2 * j + 1] << 4);
    infoA = base; infoB = 0;
    if (__ballot((all & 0x80808080u) != 0x80808080u) != 0) {
        collect_escapes(U, H, base, infoA, infoB);
    }
    L[0] ^= XM0; L[4] ^= XM1; N[0] ^= XM2;
    if (__ballot(min(min(L[0], L[4]), N[0]) == 0u) != 0) { infoA = 0x80u; infoB = 0; }
    return !(infoA & 0x80u);
}
__device__ __forceinline__ void patch_escape(u32 (&hb)[8], u32 e) {
    const u32 idx = e & 0x7FFu, val = (e >> 11) & 0xFFu, ln = idx & 63u, v = idx >> 6, sh = (v & 3u) * 8u, jj = v >> 2;
    const bool me = (threadIdx.x & 63u) == ln;
    const u32 keep = me ? ~(0xFFu << sh) : 0xFFFFFFFFu, ins = me ? val << sh : 0u;
#pragma unroll
    for (int j = 0; j < 8; ++j) hb[j] = jj == (u32)j ? (hb[j] & keep) | ins : hb[j];
}
__device__ __forceinline__ void decode(u32 (&L)[8], u32 (&N)[4], u32 infoA, u32 infoB, u32 (&d)[16]) {
    L[0] ^= XM0; L[4] ^= XM1; N[0] ^= XM2;
    const u32 rep = (infoA & 0x7Fu) * 0x01010101u;
    u32 hb[8];
#pragma unroll
    for (int j = 0; j < 4; ++j) {
        hb[2 * j] = ((N[j] & 0x07070707u) + rep) | ((N[j] & 0x08080808u) << 4);
        hb[2 * j + 1] = (((N[j] >> 4) & 0x07070707u) + rep) | (N[j] & 0x80808080u);
    }
    const u32 n = (infoA >> 8) & 3u;
    if (n) { patch_escape(hb, infoA >> 10); if (n > 1) patch_escape(hb, infoB); }
#pragma unroll
    for (int j = 0; j < 8; ++j) { d[2 * j] = perm(hb[j], L[j], 0x05010400u); d[2 * j + 1] = perm(hb[j], L[j], 0x07030602u); }
}
__device__ __forceinline__ void clear_slot(const char* base, u32 voff) {
    const u32x4 zero = {0u, 0u, 0u, 0u};
    st_sys<0>(base, voff, zero); st_sys<1024>(base, voff, zero); st_sys<2048>(base, voff, zero);
}
__device__ __forceinline__ void load_chunk(Ctx& cx, const char* base, u32 infoA, u32 infoB, int lane, u32 (&d)[16], const char* hn, u32x4& h) {
    u32x4 a, b, c, e; const u32 voff = (u32)lane * 16u, z = 0;
    if (infoA & 0x80u) {
        asm volatile("s_nop 4\n\tglobal_load_dwordx4 %0, %5, %6 sc0 sc1\n\tglobal_load_dwordx4 %1, %5, %6 offset:1024 sc0 sc1\n\t"
                     "global_load_dwordx4 %2, %5, %6 offset:2048 sc0 sc1\n\tglobal_load_dwordx4 %3, %5, %6 offset:3072 sc0 sc1\n\t"
                     "global_load_dwordx4 %4, %7, %8 sc0 sc1\n\ts_waitcnt vmcnt(0)"
                     : "=&v"(a), "=&v"(b), "=&v"(c), "=&v"(e), "=&v"(h) : "v"(voff), "s"(uni(base)), "v"(z), "s"(uni(hn)) : "memory");
        d[0] = a.x; d[1] = a.y; d[2] = a.z; d[3] = a.w; d[4] = b.x; d[5] = b.y; d[6] = b.z; d[7] = b.w;
        d[8] = c.x; d[9] = c.y; d[10] = c.z; d[11] = c.w; d[12] = e.x; d[13] = e.y; d[14] = e.z; d[15] = e.w;
        if (!GR_INACK) clear_slot(base, voff);
    } else {
        const u64 t0 = rt(); u32 polls = 0;
        for (;;) {
            asm volatile("s_nop 4\n\tglobal_load_dwordx4 %0, %4, %5 sc0 sc1\n\tglobal_load_dwordx4 %1, %4, %5 offset:1024 sc0 sc1\n\t"
                         "global_load_dwordx4 %2, %4, %5 offset:2048 sc0 sc1\n\tglobal_load_dwordx4 %3, %6, %7 sc0 sc1\n\ts_waitcnt vmcnt(0)"
                         : "=&v"(a), "=&v"(b), "=&v"(c), "=&v"(h) : "v"(voff), "s"(uni(base)), "v"(z), "s"(uni(hn)) : "memory");
            if (GR_INACK || arrived(a, b, c) || cx.dead || give_up(cx, t0, polls)) break;
        }
        clear_slot(base, voff);
        u32 L[8] = {a.x, a.y, a.z, a.w, b.x, b.y, b.z, b.w};
        u32 N[4] = {c.x, c.y, c.z, c.w};
        decode(L, N, infoA, infoB, d);
    }
}

struct InItem { int isx, s, blk, j; };
#define GR_ORDER 10
#define GR_XMASK 0x1B
#define GR_VTAB 0x31232010u
#define GR_WFULL 0
#define GR_ROT 1
__device__ __forceinline__ InItem in_item(int pos, int rel) {
    InItem it;
    it.j = pos & 3;
    const int b8 = ((pos >> 2) + (GR_ROT ? rel : 0)) & 7;
    if (pos < 256) { it.s = pos >> 6; it.isx = (pos >> 5) & 1; it.blk = b8; }
    else {
        const int rr = (pos >> 5) & 7, v = (int)((GR_VTAB >> (4 * rr)) & 3u);
        it.isx = (GR_XMASK >> rr) & 1;
        const bool full = it.isx || ((GR_WFULL >> rr) & 1);
        it.s = full ? v : (v & 1) * 2 + (b8 >> 2);
        it.blk = full ? 8 + b8 : 8 + (v >> 1) * 4 + (b8 & 3);
    }
    return it;
}
__device__ __forceinline__ void in_addr(const Params& P, int dst, const InItem& it, int lane, const char*& sbase, u32 (&vo)[4]) {
    const int dm = (dst >> 2) & 1, dn = (dst >> 1) & 1, dk = dst & 1;
    u32 stride;
    if (it.isx) {
        const int owner = (it.blk & 1) * 4 + it.j;
        const int row = owner * 512 + dm * 256 + (it.blk >> 1) * 32;
        sbase = P.x + (size_t)row * 1024 + (size_t)(dk * 256 + it.s * 64) * 2;
        stride = 1024u;
    } else {
        const int k0 = dk * 256 + it.s * 64 + (it.j & 1) * 32, n0 = dn * 2048 + it.blk * 128 + (it.j >> 1) * 64;
        sbase = P.w + (size_t)k0 * 8192 + (size_t)n0 * 2;
        stride = 8192u;
    }
    const u32 l = ((u32)lane >> 3) * stride + ((u32)lane & 7u) * 16u;
    vo[0] = l; vo[1] = l + 8u * stride; vo[2] = l + 16u * stride; vo[3] = l + 24u * stride;
}
__device__ __forceinline__ void to_regs(const u32x4 (&r)[4], u32 (&d)[16]) {
    d[0] = r[0].x; d[1] = r[0].y; d[2] = r[0].z; d[3] = r[0].w; d[4] = r[1].x; d[5] = r[1].y; d[6] = r[1].z; d[7] = r[1].w;
    d[8] = r[2].x; d[9] = r[2].y; d[10] = r[2].z; d[11] = r[2].w; d[12] = r[3].x; d[13] = r[3].y; d[14] = r[3].z; d[15] = r[3].w;
}
__device__ __forceinline__ void store_image(const char* ib, int isx, int lane, const u32 (&d)[16]) {
    const u32 r3 = (u32)lane >> 3, p = (u32)lane & 7u;
    const u32 s0 = isx ? (r3 >> 1) : 2u * ((r3 >> 1) & 1u);
    const u32 v0 = r3 * 128u + ((p ^ s0) << 4), v1 = v0 ^ 64u;
    st_agt<0>(ib, v0, (u32x4){d[0], d[1], d[2], d[3]}); st_agt<1024>(ib, v1, (u32x4){d[4], d[5], d[6], d[7]});
    st_agt<2048>(ib, v0, (u32x4){d[8], d[9], d[10], d[11]}); st_agt<3072>(ib, v1, (u32x4){d[12], d[13], d[14], d[15]});
}
__device__ __forceinline__ size_t img_off(const InItem& it, int t) {
    const int j = it.isx ? it.j : (((it.j >> 1) & 1) * 2 + (it.j & 1));
    return (it.isx ? OFF_IMGA : OFF_IMGB) + ((size_t)(it.blk * NS + t) * 4 + j) * 4096;
}
__device__ __forceinline__ void put_hdr_sys(const char* hp, u32 call, u32 a, u32 b, int lane) {
    if (lane == 0) st_sys<0>(hp, 0u, (u32x4){call, a, call, b});
}
__device__ __forceinline__ void draw_issue(u32& t, const char* ctr) {
    u64 keep; u32 z = 0, one = 1;
    asm volatile("s_nop 4\n\ts_mov_b64 %1, exec\n\ts_mov_b64 exec, 1\n\tglobal_atomic_add %0, %2, %3, %4 sc0\n\ts_mov_b64 exec, %1"
                 : "=&v"(t), "=&s"(keep) : "v"(z), "v"(one), "s"(uni(ctr)) : "memory");
}
__device__ __forceinline__ void pusher(const Params& P, Ctx& c, int cw, int wv, int lane, LAS u32x4* slots) {
    const int j = cw % 7 + 1, dst = (P.rank + j) & 7, idx = (int)rfl((u32)((cw / 7) * PW + wv));
    char* myws = P.ws[P.rank];
    char* dws = P.ws[dst];
    const u32 lo = (u32)lane * 16u;
    u32x4 A[4], B[4];
    {
        const char* din = dws + OFF_IN + (size_t)P.rank * NPOS_IN * 4096;
        const char* dhin = dws + OFF_HIN + (size_t)P.rank * NPOS_IN * 16;
        const char* ctr = myws + OFF_CTL + 256 + dst * 128;
        int p0 = idx, p1 = idx + PUSH_PER_DST, p2 = idx + 2 * PUSH_PER_DST, p3 = idx + 3 * PUSH_PER_DST;
        const u32 stage0 = rfl((u32)(size_t)slots + 8192u + (u32)wv * 12288u);
        u32 bx = 0, by = 4096, bz = 8192, par = 0;
#define GR_LOADIN(BUF, POS) do { const char* sb; u32 vo[4]; in_addr(P, dst, in_item(POS, 8 - j), lane, sb, vo); \
            asm volatile("s_mov_b32 m0, %5\n\ts_nop 4\n\tglobal_load_lds_dwordx4 %0, %4 sc1\n\tglobal_load_lds_dwordx4 %1, %4 offset:1024 sc1\n\t" \
                         "global_load_lds_dwordx4 %2, %4 offset:2048 sc1\n\tglobal_load_lds_dwordx4 %3, %4 offset:3072 sc1" \
                         :: "v"(vo[0]), "v"(vo[1] - 1024u), "v"(vo[2] - 2048u), "v"(vo[3] - 3072u), "s"(uni(sb)), "s"(stage0 + (BUF)) : "memory"); } while (0)
        GR_LOADIN(bx, p0); GR_LOADIN(by, p1);
        for (int i = wv * 64 + lane; i < NSTAGED; i += PW * 64) slots[i] = (u32x4){0u, 0u, 0u, 0u};
        asm volatile("s_waitcnt lgkmcnt(0)" ::: "memory"); BAR();
        WAITV(4);
        c.call = call_number();
        const u32 tbase = (c.call - 1u) * (u32)NPOS_IN - 4u * PUSH_PER_DST;
        bool first = true;
        for (;;) {
            {
                u64 keep; u32 z = 0, one = 1;
                asm volatile("s_nop 4\n\ts_mov_b64 %0, exec\n\ts_mov_b64 exec, 1\n\ts_cmp_eq_u32 %4, 0\n\ts_cbranch_scc1 1f\n\t"
                             "global_atomic_add v162, %1, %2, %3 sc0\n\ts_branch 2f\n1:\n\tglobal_atomic_add v161, %1, %2, %3 sc0\n2:\n\ts_mov_b64 exec, %0"
                             : "=&s"(keep) : "v"(z), "v"(one), "s"(uni(ctr)), "s"(par) : "memory", "scc");
            }
            GR_LOADIN(bz, p2 < NPOS_IN ? p2 : idx);
            u32 d[16], L[8], N[4], ia, ib;
            {
                const LAS u32x4* src = (const LAS u32x4*)(size_t)(stage0 + bx) + lane;
                const u32x4 r[4] = {src[0], src[64], src[128], src[192]};
                to_regs(r, d);
            }
            CT(65536, p0);
            const char* db = din + (size_t)p0 * 4096;
            if (encode(d, L, N, ia, ib)) {
                st_sys<0>(db, lo, (u32x4){L[0], L[1], L[2], L[3]}); st_sys<1024>(db, lo, (u32x4){L[4], L[5], L[6], L[7]}); st_sys<2048>(db, lo, (u32x4){N[0], N[1], N[2], N[3]});
            } else {
                st_sys<0>(db, lo, (u32x4){d[0], d[1], d[2], d[3]}); st_sys<1024>(db, lo, (u32x4){d[4], d[5], d[6], d[7]});
                st_sys<2048>(db, lo, (u32x4){d[8], d[9], d[10], d[11]}); st_sys<3072>(db, lo, (u32x4){d[12], d[13], d[14], d[15]});
                WAITV(0);
            }
            put_hdr_sys(dhin + (size_t)p0 * 16, c.call, ia, ib, lane);
            CT(69632, p0);
            if (first) { if (!GR_INACK) WAITV(9); first = false; p0 = p1; p1 = p2; p2 = p3; p3 = NPOS_IN; par = 1; }
            else {
                if (!GR_INACK) WAITV(13);
                u32 tk;
                asm volatile("s_cmp_eq_u32 %1, 0\n\ts_cbranch_scc1 1f\n\tv_readfirstlane_b32 %0, v161\n\ts_branch 2f\n1:\n\tv_readfirstlane_b32 %0, v162\n2:"
                             : "=&s"(tk) : "s"(par) : "memory", "scc");
                p0 = p1; p1 = p2; p2 = (int)(tk - tbase); par ^= 1u;
            }
            const u32 bt = bx; bx = by; by = bz; bz = bt;
            if (p0 >= NPOS_IN) { WAITV(0); break; }
        }
        STAMP(3);
#undef GR_LOADIN
    }
    {
        constexpr int NPOS = NSTAGED;
        const char* dp = dws + OFF_P + (size_t)P.rank * NPOS_OUT * 4096;
        const char* dhp = dws + OFF_HP + (size_t)P.rank * NPOS_OUT * 16;
        const char* q = myws + OFF_Q + (size_t)dst * NPOS_OUT * 4096;
        const char* ctr = myws + OFF_CTL + 3072 + dst * 128;
        const u32 tbase = (c.call - 1u) * (u32)NPOS - 2u * PUSH_PER_DST;
        const u32 slot0 = (u32)(size_t)slots;
        int pos = idx, nxt = idx + PUSH_PER_DST;
        u32 hA = 0, hB = 0;
        bool loaded = false;
        for (;;) {
            if (!loaded) {
                const u64 t0 = rt(); u32 polls = 0; bool ok = false;
                while (!c.dead) {
                    const u32x4 s = lds_poll16(slot0 + (u32)pos * 16u);
                    if (rfl(s.x) >= c.call && rfl(s.z) >= c.call) { hA = rfl(s.y); hB = rfl(s.w); ok = true; break; }
                    if (give_up(c, t0, polls)) break;
                }
                if (!ok) break;
                ld4_issue(A, q + (size_t)pos * 4096, lo, lo + 1024u, lo + 2048u, lo + 3072u);
                WAITV(0); PIN4(A);
            }
            const bool have_next = nxt < NPOS;
            bool ready = have_next;
            u32 nA = 0, nB = 0;
            if (have_next) {
                const u32x4 s = lds_poll16(slot0 + (u32)nxt * 16u);
                ready = (rfl(s.x) >= c.call && rfl(s.z) >= c.call);
                nA = rfl(s.y); nB = rfl(s.w);
            }
            u32 tk;
            draw_issue(tk, ctr);
            ld4_issue(B, q + (size_t)(ready ? nxt : pos) * 4096, lo, lo + 1024u, lo + 2048u, lo + 3072u);
            CT(65536, NPOS_IN + pos);
            const char* db = dp + (size_t)pos * 4096; const char* hp = dhp + (size_t)pos * 16;
            st_sys<0>(db, lo, A[0]); st_sys<1024>(db, lo, A[1]); st_sys<2048>(db, lo, A[2]);
            if (hA & 0x80u) { st_sys<3072>(db, lo, A[3]); WAITV(0); }
            put_hdr_sys(hp, c.call, hA, hB, lane);
            CT(69632, NPOS_IN + pos);
            int nn;
            if (ready) {
                WAITV(4);
                PIN4(B); asm volatile("" : "+v"(tk));
                nn = (int)(rfl(tk) - tbase);
                A[0] = B[0]; A[1] = B[1]; A[2] = B[2]; A[3] = B[3];
                hA = nA; hB = nB; loaded = true;
            } else {
                WAITV(0); PIN4(B); asm volatile("" : "+v"(tk));
                nn = (int)(rfl(tk) - tbase);
                loaded = false;
                if (!have_next) break;
            }
            pos = nxt; nxt = nn;
        }
    }
    STAMP(2);
}
__device__ __forceinline__ void scout(const Params& P, Ctx& c, int cw, int lane, LAS u32x4* slots) {
    static_assert(DIRECT_B, "only the 128 headers of tile A are forwarded");
    const int j = cw % 7 + 1, dst = (P.rank + j) & 7;
    const char* hq = P.ws[P.rank] + OFF_HQ + (size_t)dst * NPOS_OUT * 16;
    const u32 vo = (u32)lane * 16u;
    u32 done = NSTAGED == 64 ? 2u : 0u;
    const u64 t0 = rt(); u32 polls = 0;
#define GR_SCOUTSLEEP 8
    for (;;) {
        u32x4 h0, h1;
        asm volatile("s_nop 4\n\tglobal_load_dwordx4 %0, %2, %3 sc1\n\tglobal_load_dwordx4 %1, %2, %3 offset:1024 sc1\n\ts_waitcnt vmcnt(0)"
                     : "=&v"(h0), "=&v"(h1) : "v"(vo), "s"(uni(hq)) : "memory");
        if (!(done & 1u) && h0.x >= c.call && h0.z >= c.call) { slots[lane] = h0; done |= 1u; }
        if (!(done & 2u) && h1.x >= c.call && h1.z >= c.call) { slots[64 + lane] = h1; done |= 2u; }
        if (__all(done == 3u)) break;
        if (!__any((done & 1u) != 0u)) __builtin_amdgcn_s_sleep(32); else __builtin_amdgcn_s_sleep(GR_SCOUTSLEEP);
        if (give_up(c, t0, polls)) break;
    }
}
static_assert((NPOS_IN / 4 + NQUAD - 1) / NQUAD + (NPOS_IN / 4 + NQUAD / 7 - 1) / (NQUAD / 7) <= 32, "LDS slots per quad");
#define GR_TKT 1
struct Done { u32 slot, line, t, k; };
__device__ __forceinline__ bool group_done(char* myws, const Done& d, u32 call, LAS u32* cnt, LAS u32* tick, int lane) {
    u32 old = 0;
    if (lane == 0) old = __hip_atomic_fetch_add(cnt + d.slot, 1u, __ATOMIC_RELAXED, __HIP_MEMORY_SCOPE_WORKGROUP);
    if (rfl(old) == 3u) {
        u32 k = d.k;
        if (k > 31u) k = rfl(__hip_atomic_load(tick + d.slot, __ATOMIC_RELAXED, __HIP_MEMORY_SCOPE_WORKGROUP)) & 31u;
        const u32 off = lane == 0 ? (u32)OFF_FLG + d.line * 128u + d.t * 4u : (u32)OFF_LOG + d.line * 128u + k * 4u;
        if (lane < 2) __hip_atomic_store((GAS u32*)(myws + off), lane == 0 ? call : (call << 5) | d.t, __ATOMIC_RELAXED, __HIP_MEMORY_SCOPE_AGENT);
        return true;
    }
    return false;
}
__device__ __forceinline__ void unpacker(const Params& P, Ctx& c, int quad, int j, int lane, LAS u32* cnt, LAS u32* tick, u32 pbuf) {
    char* myws = P.ws[P.rank];
    Done pd; bool pend = false;
    u32 slot = 0;
    for (int h = quad; h < NPOS_IN / 4; h += NQUAD, ++slot) {
        const InItem it = in_item(h * 4 + j, 0); const char* sb; u32 vv[4]; u32x4 r[4]; u32 d[16];
        in_addr(P, P.rank, it, lane, sb, vv);
        ld4_issue(r, sb, vv[0], vv[1], vv[2], vv[3]);
        WAITV(0); PIN4(r);
        if (pend) group_done(myws, pd, c.call, cnt, tick, lane);
        to_regs(r, d);
        store_image(myws + img_off(it, it.s), it.isx, lane, d);
        pd.slot = slot; pd.line = (u32)((it.isx ? 0 : 16) + it.blk); pd.t = (u32)it.s; pd.k = (u32)it.s; pend = true;
    }
    if (threadIdx.x < 512) STAMP(1);
    constexpr int QPS = NQUAD / 7;
    const int rel = quad % 7 + 1;
#define GR_UPIPE 0
    u32x4 hn = {0u, 0u, 0u, 0u};
    const int src = (P.rank + rel) & 7;
    for (int g = quad < QPS * 7 ? quad / 7 : NPOS_IN; g < NPOS_IN / 4; g += QPS, ++slot) {
        const int pos = g * 4 + j;
        const char* hp = myws + OFF_HIN + ((size_t)src * NPOS_IN + pos) * 16;
        u32 ia = 0, ib2 = 0; bool ok = false;
        if (rfl(hn.x) >= c.call && rfl(hn.z) >= c.call) { ia = rfl(hn.y); ib2 = rfl(hn.w); ok = true; CT(81920, q); }
        else if (!c.dead) {
            const u64 t0 = rt(); u32 polls = 0;
            for (;;) {
                const u32x4 h = poll16(hp);
                if (pend) { group_done(myws, pd, c.call, cnt, tick, lane); pend = false; CT(98304, prevq); }
                if (rfl(h.x) >= c.call && rfl(h.z) >= c.call) { ia = rfl(h.y); ib2 = rfl(h.w); ok = true; CT(81920, q); break; }
                if (give_up(c, t0, polls)) break;
            }
        }
        if (!ok) break;
        const InItem it = in_item(pos, rel);
        const int t = 4 + it.s * 7 + (rel - 1);
        const u32 line = (u32)((it.isx ? 0 : 16) + it.blk);
        u32 tk = 0;
        bool draw = GR_TKT == 0 && j == 0;
        if (GR_TKT == 1) {
            u32 old = 0;
            if (lane == 0) old = __hip_atomic_fetch_add(tick + 32 + slot, 1u, __ATOMIC_RELAXED, __HIP_MEMORY_SCOPE_WORKGROUP);
            draw = rfl(old) == 3u;
        }
        if (draw) draw_issue(tk, myws + OFF_TKT + line * 128u);
        u32 d[16];
        load_chunk(c, myws + OFF_IN + ((size_t)src * NPOS_IN + pos) * 4096, ia, ib2, lane, d, g + QPS < NPOS_IN / 4 ? hp + QPS * 4 * 16 : hp, hn);
        if (g + QPS >= NPOS_IN / 4) hn = (u32x4){0u, 0u, 0u, 0u};
        if (pend) { group_done(myws, pd, c.call, cnt, tick, lane); pend = false; CT(98304, prevq); }
        if (draw) {
            asm volatile("" : "+v"(tk));
            if (lane == 0) __hip_atomic_store(tick + slot, 4u + (tk - (c.call - 1u) * 28u), __ATOMIC_RELAXED, __HIP_MEMORY_SCOPE_WORKGROUP);
        }
        store_image(myws + img_off(it, t), it.isx, lane, d);
        pd.slot = slot; pd.line = line; pd.t = (u32)t; pd.k = 32u; pend = true;
    }
    WAITV(0);
    CT(98304, prevq);
    if (pend && !c.dead) group_done(myws, pd, c.call, cnt, tick, lane);
    if (threadIdx.x < 512) STAMP(2);
}

struct Frags { s16x8 a[2][2]; s16x4 bl[4][2], bh[4][2]; };
template <int OFF> __device__ __forceinline__ void dsr128(s16x8& r, u32 addr) { asm volatile("ds_read_b128 %0, %1 offset:%2" : "=v"(r) : "v"(addr), "i"(OFF)); }
template <int OFF> __device__ __forceinline__ void dsrtr(s16x4& r, u32 addr) { asm volatile("ds_read_b64_tr_b16 %0, %1 offset:%2" : "=v"(r) : "v"(addr), "i"(OFF)); }
__device__ __forceinline__ void mf(f32x4& acc, const s16x4& bl, const s16x4& bh, const s16x8& a) {
    s16x8 b = __builtin_shufflevector(bl, bh, 0, 1, 2, 3, 4, 5, 6, 7);
    asm volatile("v_mfma_f32_16x16x32_bf16 %0, %1, %2, %0" : "+v"(acc) : "v"(b), "v"(a));
}
template <int OFF> __device__ __forceinline__ void dma(const char* sbase, u32 voff, LAS char* dst) {
    __builtin_amdgcn_global_load_lds((const GAS void*)(sbase + voff), (LAS void*)dst, 16, OFF, 16  );
}
__device__ __forceinline__ void wait_frags(Frags& f) {
    asm volatile("s_waitcnt lgkmcnt(0)" : "+v"(f.a[0][0]), "+v"(f.a[0][1]), "+v"(f.a[1][0]), "+v"(f.a[1][1]),
        "+v"(f.bl[0][0]), "+v"(f.bl[1][0]), "+v"(f.bl[2][0]), "+v"(f.bl[3][0]), "+v"(f.bl[0][1]), "+v"(f.bl[1][1]), "+v"(f.bl[2][1]), "+v"(f.bl[3][1]) :: "memory");
    asm volatile("" : "+v"(f.bh[0][0]), "+v"(f.bh[1][0]), "+v"(f.bh[2][0]), "+v"(f.bh[3][0]), "+v"(f.bh[0][1]), "+v"(f.bh[1][1]), "+v"(f.bh[2][1]), "+v"(f.bh[3][1]));
}
struct FAddr { u32 a[2], b[4]; };
#define RA(mi, kk) dsr128<mi * 2048>(n.a[mi][kk], ad.a[kk] + so)
#define RB(ni, kk) do { dsrtr<kk * 4096>(n.bl[ni][kk], ad.b[ni] + so); dsrtr<kk * 4096 + 512>(n.bh[ni][kk], ad.b[ni] + so); } while (0)
__device__ __forceinline__ void read_all(Frags& n, const FAddr& ad, u32 so) {
    RA(0,0); RA(1,0); RB(0,0); RB(1,0); RB(2,0); RB(3,0); RA(0,1); RA(1,1); RB(0,1); RB(1,1); RB(2,1); RB(3,1);
}
#define MF(mi, ni, kk) mf(acc[mi][ni], c.bl[ni][kk], c.bh[ni][kk], c.a[mi][kk])
__device__ __forceinline__ void stage_addr(FAddr& ad, const FAddr& ad0, u32 so) {
    ad.a[0] = ad0.a[0] + so; ad.a[1] = ad0.a[1] + so; ad.b[0] = ad0.b[0] + so; ad.b[1] = ad0.b[1] + so; ad.b[2] = ad0.b[2] + so; ad.b[3] = ad0.b[3] + so;
    asm volatile("" : "+v"(ad.a[0]), "+v"(ad.a[1]), "+v"(ad.b[0]), "+v"(ad.b[1]), "+v"(ad.b[2]), "+v"(ad.b[3]));
}
__device__ __forceinline__ u32 ring_next(u32 b) { return b + 32768 == NST * 32768 ? 0 : b + 32768; }
#define GR_KV 0
template <int VAR> __device__ __forceinline__ void kstep(f32x4 (&acc)[2][4], const Frags& c, Frags& n, FAddr& ad, const FAddr& ad0, u32 son) {
#undef RA
#undef RB
#define RA(mi, kk) dsr128<mi * 2048>(n.a[mi][kk], ad.a[kk])
#define RB(ni, kk) do { dsrtr<kk * 4096>(n.bl[ni][kk], ad.b[ni]); dsrtr<kk * 4096 + 512>(n.bh[ni][kk], ad.b[ni]); } while (0)
    if (VAR == 0) {
        MF(0,0,0); RA(0,0); RA(1,0);
        MF(0,1,0); RB(0,0);
        MF(0,2,0); RB(1,0);
        MF(0,3,0); RB(2,0);
        MF(1,0,0); RB(3,0);
        MF(1,1,0); RA(0,1); RA(1,1);
        MF(1,2,0); RB(0,1);
        MF(1,3,0); RB(1,1);
        MF(0,0,1); RB(2,1);
        MF(0,1,1); RB(3,1);
        MF(0,2,1);
        stage_addr(ad, ad0, son);
        MF(0,3,1);
        MF(1,0,1);
        MF(1,1,1);
        MF(1,2,1);
        MF(1,3,1);
    } else if (VAR == 1) {
        MF(0,0,0);
        MF(0,1,0);
        MF(0,2,0);
        MF(0,3,0);
        MF(1,0,0); RA(0,0); RA(1,0);
        MF(1,1,0); RB(0,0);
        MF(1,2,0); RB(1,0);
        MF(1,3,0); RB(2,0);
        MF(0,0,1); RB(3,0);
        MF(0,1,1); RA(0,1); RA(1,1);
        MF(0,2,1); RB(0,1);
        MF(0,3,1); RB(1,1);
        MF(1,0,1); RB(2,1);
        MF(1,1,1); RB(3,1);
        stage_addr(ad, ad0, son);
        MF(1,2,1);
        MF(1,3,1);
    } else if (VAR == 2) {
        MF(0,0,0); RA(0,0);
        MF(0,1,0); RA(1,0);
        MF(0,2,0); RB(0,0);
        MF(0,3,0); RB(1,0);
        MF(1,0,0); RB(2,0);
        MF(1,1,0); RB(3,0);
        MF(1,2,0); RA(0,1);
        MF(1,3,0); RA(1,1);
        MF(0,0,1); RB(0,1);
        MF(0,1,1); RB(1,1);
        MF(0,2,1); RB(2,1);
        MF(0,3,1); RB(3,1);
        MF(1,0,1);
        stage_addr(ad, ad0, son);
        MF(1,1,1);
        MF(1,2,1);
        MF(1,3,1);
    } else {
        MF(0,0,0);
        MF(0,1,0);
        MF(0,2,0);
        MF(0,3,0);
        MF(1,0,0);
        MF(1,1,0);
        MF(1,2,0); RA(0,0); RA(1,0);
        MF(1,3,0); RB(0,0);
        MF(0,0,1); RB(1,0);
        MF(0,1,1); RB(2,0);
        MF(0,2,1); RB(3,0);
        MF(0,3,1); RA(0,1); RA(1,1);
        MF(1,0,1); RB(0,1);
        MF(1,1,1); RB(1,1);
        MF(1,2,1); RB(2,1);
        MF(1,3,1); RB(3,1);
        stage_addr(ad, ad0, son);
    }
}
#undef MF
#undef RA
#undef RB
__device__ __forceinline__ void dma_chunk(const char* sbase, u32 voff, u32 m0v) {
    asm volatile("s_mov_b32 m0, %2\n\ts_nop 0\n\tglobal_load_lds_dwordx4 %0, %1 sc1\n\tglobal_load_lds_dwordx4 %0, %1 offset:1024 sc1\n\t"
                 "global_load_lds_dwordx4 %0, %1 offset:2048 sc1\n\tglobal_load_lds_dwordx4 %0, %1 offset:3072 sc1" :: "v"(voff), "s"(sbase), "s"(m0v) : "memory");
}
constexpr u32 AV_ALL = 0x7FFFFFFFu, AV_DEAD = 0x80000000u;
__device__ __forceinline__ u32 avail_of(u32 d, u32 f, u32 call, u32 ulast) {
    const u64 fm = (u64)__ballot(f >= call);
    const u32 bit = ((threadIdx.x & 32u) | (d & 31u));
    const bool ok = (d >> 5) >= call && ((fm >> bit) & 1ull) != 0ull;
    const u64 miss = ~(u64)__ballot(ok);
    const u32 a = miss ? (u32)__builtin_ctzll(miss) : 64u;
    return a > ulast ? AV_ALL : a;
}
#define GR_READ_LINES(D, F) asm volatile("v_mov_b32 %0, v160\n\tv_mov_b32 %1, v161" : "=v"(D), "=v"(F))
__device__ __forceinline__ u32 wait_avail(const char* tmo, u32 call, const char* base, u32 dvo, u32 cvo, u32 ulast, u32 u) {
    Ctx c; c.tmo = tmo; c.call = call; c.dead = false;
    const u64 t0 = rt(); u32 polls = 0;
    for (;;) {
        asm volatile("s_nop 4\n\tglobal_load_dword v160, %0, %2 sc1\n\tglobal_load_dword v161, %1, %2 sc1\n\ts_waitcnt vmcnt(0)" :: "v"(dvo), "v"(cvo), "s"(uni(base)) : "memory", "v160", "v161");
        u32 d, f; GR_READ_LINES(d, f);
        const u32 av = avail_of(d, f, call, ulast);
        if (av > u) return av;
        if (give_up(c, t0, polls)) return AV_DEAD;
    }
}
__device__ __forceinline__ u32 cvt_pk_bf16(float lo, float hi) { u32 r; asm volatile("v_cvt_pk_bf16_f32 %0, %1, %2" : "=v"(r) : "v"(lo), "v"(hi)); return r; }
template <int N> __device__ __forceinline__ void waitv() { asm volatile("s_waitcnt vmcnt(%0)" :: "i"(N) : "memory"); }

__device__ __forceinline__ float bf_lo(u32 v) { return __uint_as_float(v << 16); }
__device__ __forceinline__ float bf_hi(u32 v) { return __uint_as_float(v & 0xFFFF0000u); }

#define GR_TRACE(u) do { } while (0)
#define WT0() do { } while (0)
#define WT1() do { } while (0)
__device__ __forceinline__ void gemm_loader(const Params& P, Ctx& c, LAS char* lds, int rb, int cb0, int ntile, int l, int lane) {
    char* myws = P.ws[P.rank];
    const char* srcx = uni(myws + OFF_IMGA + ((size_t)rb * NS * 4 + l) * 4096);
    const char* srcw0 = uni(myws + OFF_IMGB + ((size_t)cb0 * NS * 4 + l) * 4096);
    const char* srcw1 = ntile == 1 ? srcw0 : srcw0 + (size_t)8 * NS * 4 * 4096;
    const char* lines = uni(myws + OFF_LOG);
    u32 dvo, cvo;
    {
        const int cbl = (ntile == 2 && lane >= 32) ? cb0 + 8 : cb0;
        const bool isb = cbl >= 8;
        dvo = (u32)(isb ? 16 + cbl : rb) * 128u + ((u32)lane & 31u) * 4u;
        cvo = 4096u + (u32)(isb ? rb : 16 + cbl) * 128u + ((u32)lane & 31u) * 4u;
    }
    const u32 ulast = (u32)(ntile * NS - 1), voff = (u32)lane * 16u;
    const u32 ldsw0 = (u32)(size_t)lds + (u32)l * 4096u;
    u32 avail = 0, iss = 0, bi = 0;
    asm volatile("v_mov_b32 v160, 0\n\tv_mov_b32 v161, 0" ::: "v160", "v161");
#define GR_POLL() asm volatile("global_load_dword v160, %0, %2 sc1\n\tglobal_load_dword v161, %1, %2 sc1" :: "v"(dvo), "v"(cvo), "s"(lines) : "memory", "v160", "v161")
#define GR_SNAP() do { u32 _d, _f; GR_READ_LINES(_d, _f); avail = max(avail, avail_of(_d, _f, c.call, ulast)); } while (0)
#define GR_ISSUE_ONE() do { \
        u32 _e; asm volatile("v_readlane_b32 %0, v160, %1" : "=s"(_e) : "s"(iss));         \
        const u32 _off = (_e << 14) & 0x7C000u; \
        dma_chunk(srcx + _off, voff, ldsw0 + bi); dma_chunk(((iss & NS) ? srcw1 : srcw0) + _off, voff, ldsw0 + bi + 16384u); \
        ++iss; bi = ring_next(bi); } while (0)
#define GR_ISSUE_UPTO(LIM) do { const u32 _lim = min(avail, (u32)(LIM)); while (iss < _lim) GR_ISSUE_ONE(); } while (0)
#define GR_WAITK(K) do { const int _k = (K); if (_k >= 4) waitv<32>(); else if (_k == 3) waitv<24>(); else if (_k == 2) waitv<16>(); else if (_k == 1) waitv<8>(); else waitv<0>(); } while (0)
    static_assert(NST == 5, "GR_WAITK covers NST - 1 = 4 younger stages");
    avail = wait_avail(c.tmo, c.call, lines, dvo, cvo, ulast, 0u);
    GR_ISSUE_UPTO(NST);
    GR_WAITK((int)iss - 1);
    BAR();
    const u32 nstage = ulast + 1, call5 = c.call << 5;
    const u32 xlo = (u32)(u64)srcx, xhi = (u32)((u64)srcx >> 32), w0lo = (u32)(u64)srcw0, w0hi = (u32)((u64)srcw0 >> 32), w1lo = (u32)(u64)srcw1, w1hi = (u32)((u64)srcw1 >> 32);
#define GR_LSTEP(PARITY) do { \
        if (__builtin_expect(iss <= u + 1 && u < ulast, 0)) {          \
            WT0(); const u32 a = wait_avail(c.tmo, c.call, lines, dvo, cvo, ulast, u + 1); WT1(); \
            if (a == AV_DEAD) asm volatile("v_mov_b32 v161, -1" ::: "v161"); \
            u32 keep = max(avail, a); avail = a; \
            GR_ISSUE_UPTO(min(u + NST, nstage)); \
            if (iss < min(u + NST, nstage) && avail < AV_ALL) {        \
                GR_POLL(); WAITV(0); GR_SNAP(); keep = max(keep, avail); \
                GR_ISSUE_UPTO(min(u + NST, nstage)); \
            } \
            avail = keep; \
        } \
        u32 t0, t1, t2; \
        asm volatile( \
            "s_sub_u32 %[t0], %[iss], %[u]\n\ts_waitcnt vmcnt(24)\n\t"             \
            "s_cmp_ge_i32 %[t0], 5\n\ts_cbranch_scc1 .Lgr_bar%=\n\ts_waitcnt vmcnt(16)\n\t" \
            "s_cmp_ge_i32 %[t0], 4\n\ts_cbranch_scc1 .Lgr_bar%=\n\ts_waitcnt vmcnt(8)\n\t" \
            "s_cmp_ge_i32 %[t0], 3\n\ts_cbranch_scc1 .Lgr_bar%=\n\ts_waitcnt vmcnt(0)\n" \
            ".Lgr_bar%=:\n\ts_barrier\n\t"                                         \
            "v_cmp_le_u32_e64 s[88:89], %[call5], v160\n\tv_cmp_le_u32_e64 s[90:91], %[call], v161\n\t" \
            "s_not_b64 s[88:89], s[88:89]\n\ts_ff1_i32_b64 %[t0], s[88:89]\n\ts_min_u32 %[t0], %[t0], 64\n\t" \
            "s_cmp_gt_u32 %[t0], %[ulast]\n\ts_cselect_b32 %[t0], 0x7fffffff, %[t0]\n\ts_max_u32 %[avail], %[avail], %[t0]\n\t" \
            "s_add_u32 %[t1], %[u], 6\n\ts_min_u32 %[t1], %[t1], %[nst]\n\ts_min_u32 %[t1], %[t1], %[avail]\n" \
            ".Lgr_st%=:\n\t" \
            "v_readlane_b32 %[t0], v160, %[iss]\n\ts_and_b32 %[t2], %[iss], 32\n\t" \
            "s_cmp_lt_u32 %[iss], %[t1]\n\ts_cbranch_scc0 .Lgr_skip%=\n\t" \
            "s_and_b32 %[t0], %[t0], 31\n\ts_or_b32 %[t2], %[t2], %[t0]\n\ts_bitcmp1_b64 s[90:91], %[t2]\n\ts_cbranch_scc0 .Lgr_skip%=\n\t" \
            "s_lshl_b32 %[t0], %[t0], 14\n\ts_add_u32 m0, %[ldsw0], %[bi]\n\t" \
            "s_add_u32 s92, %[xlo], %[t0]\n\ts_addc_u32 s93, %[xhi], 0\n\t" \
            "s_bitcmp1_b32 %[iss], 5\n\ts_cselect_b32 s94, %[w1lo], %[w0lo]\n\ts_cselect_b32 s95, %[w1hi], %[w0hi]\n\t" \
            "global_load_lds_dwordx4 %[voff], s[92:93] sc1\n\tglobal_load_lds_dwordx4 %[voff], s[92:93] offset:1024 sc1\n\t" \
            "global_load_lds_dwordx4 %[voff], s[92:93] offset:2048 sc1\n\tglobal_load_lds_dwordx4 %[voff], s[92:93] offset:3072 sc1\n\t" \
            "s_add_u32 s94, s94, %[t0]\n\ts_addc_u32 s95, s95, 0\n\ts_add_u32 m0, m0, 0x4000\n\ts_add_u32 %[iss], %[iss], 1\n\t" \
            "global_load_lds_dwordx4 %[voff], s[94:95] sc1\n\tglobal_load_lds_dwordx4 %[voff], s[94:95] offset:1024 sc1\n\t" \
            "global_load_lds_dwordx4 %[voff], s[94:95] offset:2048 sc1\n\tglobal_load_lds_dwordx4 %[voff], s[94:95] offset:3072 sc1\n\t" \
            "s_add_u32 %[bi], %[bi], 0x8000\n\ts_cmp_eq_u32 %[bi], 0x28000\n\ts_cselect_b32 %[bi], 0, %[bi]\n\t" \
            "s_cmp_lt_u32 %[iss], %[t1]\n\ts_cbranch_scc1 .Lgr_st%=\n" \
            ".Lgr_skip%=:\n\t"                                                     \
            "s_cmp_lt_u32 %[iss], %[t1]\n\ts_cselect_b32 %[t2], 1, 0\n\ts_add_u32 %[t0], %[iss], 3\n\t" \
            "s_cmp_le_u32 %[avail], %[t0]\n\ts_cselect_b32 %[t2], 1, %[t2]\n\ts_and_b32 %[t2], %[t2], " #PARITY "\n\ts_cmp_lg_u32 %[t2], 0\n\ts_cselect_b64 exec, -1, 0\n\t" \
            "global_load_dword v160, %[dvo], %[lines] sc1\n\ts_andn2_b64 exec, exec, s[90:91]\n\t"          \
            "global_load_dword v161, %[cvo], %[lines] sc1\n\ts_mov_b64 exec, -1" \
            : [iss] "+s"(iss), [bi] "+s"(bi), [avail] "+s"(avail), [t0] "=&s"(t0), [t1] "=&s"(t1), [t2] "=&s"(t2) \
            : [u] "s"(u), [call5] "s"(call5), [call] "s"(c.call), [ulast] "s"(ulast), [nst] "s"(nstage), [ldsw0] "s"(ldsw0), \
              [xlo] "s"(xlo), [xhi] "s"(xhi), [w0lo] "s"(w0lo), [w0hi] "s"(w0hi), [w1lo] "s"(w1lo), [w1hi] "s"(w1hi), \
              [voff] "v"(voff), [dvo] "v"(dvo), [cvo] "v"(cvo), [lines] "s"(lines) \
            : "memory", "scc", "s88", "s89", "s90", "s91", "s92", "s93", "s94", "s95", "v160", "v161"); \
        ++u; } while (0)
    static_assert(NST == 5 && NS == 32, "constants of the loader step");
#pragma unroll 1
#define GR_LUNR 4
#define GR_POLL2 0
#define GR_LSTEP4() GR_LSTEPX(1); GR_LSTEPX(GR_POLL2); GR_LSTEPX(1); GR_LSTEPX(GR_POLL2)
#define GR_LSTEPX(P) GR_LSTEP(P)
    for (u32 u = 0; u <= ulast;) {
        GR_LSTEP4();
        if (GR_LUNR >= 8) { GR_LSTEP4(); }
        if (GR_LUNR >= 16) { GR_LSTEP4(); GR_LSTEP4(); }
        if (GR_LUNR >= 32) { GR_LSTEP4(); GR_LSTEP4(); GR_LSTEP4(); GR_LSTEP4(); }
    }
#undef GR_LSTEP4
#undef GR_LSTEPX
#undef GR_LSTEP
#undef GR_ISSUE_ONE
    WAITV(0);
    BAR();
#undef GR_POLL
#undef GR_SNAP
#undef GR_ISSUE_UPTO
#undef GR_WAITK
}
template <int VAR> __device__ __forceinline__ void gemm_mfma(const Params& P, Ctx& c, LAS char* lds, int rb, int cb0, int ntile, int w, int lane) {
    char* myws = P.ws[P.rank];
    const int wm = w & 3, wn = w >> 2;
    const u32 ldsbase = (u32)(size_t)lds;
    FAddr ad;
    {
        const u32 r = (u32)lane & 15u, g = (u32)lane >> 4;
        ad.a[0] = ldsbase + wm * 4096 + r * 128u + ((g ^ (r >> 1)) << 4);
        ad.a[1] = ad.a[0] ^ 64u;
        const u32 q = r >> 2, pp = r & 3u, sw = (q >> 1) + 2u * (g & 1u);
#pragma unroll
        for (int ni = 0; ni < 4; ++ni) ad.b[ni] = ldsbase + 16384 + wn * 8192 + (8u * g + q) * 128u + (((u32)ni ^ sw) << 5) + pp * 8u;
    }
    f32x4 acc[2][4];
#pragma unroll
    for (int mi = 0; mi < 2; ++mi)
#pragma unroll
        for (int ni = 0; ni < 4; ++ni) acc[mi][ni] = (f32x4){0.f, 0.f, 0.f, 0.f};
    Frags f0, f1;
    BAR();
    read_all(f0, ad, 0u);
    wait_frags(f0);
    asm volatile("s_nop 1");
    u32 brn = 2 * 32768;
    const char* pend_h = nullptr; u32 pendA = 0, pendB = 0; bool pend = false;
    FAddr adr; stage_addr(adr, ad, 32768u);
    for (int tile = 0; tile < ntile; ++tile) {
        const int cb = cb0 + 8 * tile;
#define GR_UNR 8
#pragma unroll 1
        for (int s = 0; s < NS; s += GR_UNR) {
#define GR_STEP(S, CUR, NXT) do { \
            BAR(); SCHED(); \
            GR_TRACE((u32)(tile * NS + (S))); \
            kstep<VAR>(acc, CUR, NXT, adr, ad, brn); \
            brn = ring_next(brn); \
            SCHED(); \
            wait_frags(NXT); SCHED(); } while (0)
            GR_STEP(s, f0, f1);
            GR_STEP(s + 1, f1, f0);
            if (__builtin_expect(tile == 1 && s == 0, 0)) { WAITV(0); if (pend) put_hdr_sys(pend_h, c.call, pendA, pendB, lane); }
            if (GR_UNR >= 4) { GR_STEP(s + 2, f0, f1); GR_STEP(s + 3, f1, f0); }
            if (GR_UNR >= 8) { GR_STEP(s + 4, f0, f1); GR_STEP(s + 5, f1, f0); GR_STEP(s + 6, f0, f1); GR_STEP(s + 7, f1, f0); }
            if (GR_UNR >= 16) { GR_STEP(s + 8, f0, f1); GR_STEP(s + 9, f1, f0); GR_STEP(s + 10, f0, f1); GR_STEP(s + 11, f1, f0); GR_STEP(s + 12, f0, f1); GR_STEP(s + 13, f1, f0); GR_STEP(s + 14, f0, f1); GR_STEP(s + 15, f1, f0); }
            if (GR_UNR >= 32) { GR_STEP(s + 16, f0, f1); GR_STEP(s + 17, f1, f0); GR_STEP(s + 18, f0, f1); GR_STEP(s + 19, f1, f0); GR_STEP(s + 20, f0, f1); GR_STEP(s + 21, f1, f0); GR_STEP(s + 22, f0, f1); GR_STEP(s + 23, f1, f0);
                                GR_STEP(s + 24, f0, f1); GR_STEP(s + 25, f1, f0); GR_STEP(s + 26, f0, f1); GR_STEP(s + 27, f1, f0); GR_STEP(s + 28, f0, f1); GR_STEP(s + 29, f1, f0); GR_STEP(s + 30, f0, f1); GR_STEP(s + 31, f1, f0); }
#undef GR_STEP
        }
        asm volatile("s_nop 15\n\ts_nop 15" ::: "memory");
        asm volatile("" : "+v"(acc[0][0]), "+v"(acc[0][1]), "+v"(acc[0][2]), "+v"(acc[0][3]), "+v"(acc[1][0]), "+v"(acc[1][1]), "+v"(acc[1][2]), "+v"(acc[1][3]));
        u32 d[16], L[8], N[4], ia, ib;
#pragma unroll
        for (int mi = 0; mi < 2; ++mi)
#pragma unroll
            for (int ni = 0; ni < 4; ++ni) {
                d[mi * 8 + ni * 2] = cvt_pk_bf16(acc[mi][ni][0], acc[mi][ni][1]);
                d[mi * 8 + ni * 2 + 1] = cvt_pk_bf16(acc[mi][ni][2], acc[mi][ni][3]);
                acc[mi][ni] = (f32x4){0.f, 0.f, 0.f, 0.f};
            }
        const int owner = (rb & 1) * 4 + wm;
        const int cidx = (cb >> 3) * 128 + (rb >> 1) * 16 + (cb & 7) * 2 + wn;
        const bool mine = owner == P.rank;
        const u32 lo = (u32)lane * 16u;
        const bool direct = cidx >= NSTAGED;
        char* tws = (direct || mine) ? P.ws[owner] : myws;
        const int slot = ((direct || mine) ? P.rank : owner) * NPOS_OUT + cidx;
        const char* db = tws + ((direct || mine) ? OFF_P : OFF_Q) + (size_t)slot * 4096;
        const char* hp = tws + ((direct || mine) ? OFF_HP : OFF_HQ) + (size_t)slot * 16;
        const bool packed = encode(d, L, N, ia, ib);
        st_sys<0>(db, lo, packed ? (u32x4){L[0], L[1], L[2], L[3]} : (u32x4){d[0], d[1], d[2], d[3]});
        st_sys<1024>(db, lo, packed ? (u32x4){L[4], L[5], L[6], L[7]} : (u32x4){d[4], d[5], d[6], d[7]});
        st_sys<2048>(db, lo, packed ? (u32x4){N[0], N[1], N[2], N[3]} : (u32x4){d[8], d[9], d[10], d[11]});
        if (!packed) st_sys<3072>(db, lo, (u32x4){d[12], d[13], d[14], d[15]});
        pend = !(packed && (direct || mine));
        if (!pend) put_hdr_sys(hp, c.call, ia, ib, lane);
        pend_h = hp; pendA = ia; pendB = ib;
    }
    if (pend) { WAITV(0); put_hdr_sys(pend_h, c.call, pendA, pendB, lane); }
    BAR();
}

__device__ __forceinline__ void unpack_regs(const u32x4& a, const u32x4& b, const u32x4& cc, const u32x4& e, u32 infoA, u32 infoB, int lane, u32 (&d)[16]) {
    if (infoA & 0x80u) {
        d[0] = a.x; d[1] = a.y; d[2] = a.z; d[3] = a.w; d[4] = b.x; d[5] = b.y; d[6] = b.z; d[7] = b.w;
        d[8] = cc.x; d[9] = cc.y; d[10] = cc.z; d[11] = cc.w; d[12] = e.x; d[13] = e.y; d[14] = e.z; d[15] = e.w;
    } else {
        u32 L[8] = {a.x, a.y, a.z, a.w, b.x, b.y, b.z, b.w};
        u32 N[4] = {cc.x, cc.y, cc.z, cc.w};
        decode(L, N, infoA, infoB, d);
    }
}

__device__ __forceinline__ bool poll_all(Ctx& c, const char* base, u32 vo, u32x4& h) {
    const char* sb = uni(base);
    const u64 t0 = rt(); u32 polls = 0;
    asm volatile("s_nop 4\n\tglobal_load_dwordx4 v[160:163], %0, %1 sc0 sc1" :: "v"(vo), "s"(sb) : "memory", "v160", "v161", "v162", "v163");
    for (;;) {
        asm volatile("global_load_dwordx4 v[164:167], %4, %5 sc0 sc1\n\ts_waitcnt vmcnt(1)\n\tv_mov_b32 %0, v160\n\tv_mov_b32 %1, v161\n\tv_mov_b32 %2, v162\n\tv_mov_b32 %3, v163"
                     : "=&v"(h.x), "=&v"(h.y), "=&v"(h.z), "=&v"(h.w) : "v"(vo), "s"(sb) : "memory", "v164", "v165", "v166", "v167");
        if (__all(h.x >= c.call && h.z >= c.call)) return true;
        asm volatile("global_load_dwordx4 v[160:163], %4, %5 sc0 sc1\n\ts_waitcnt vmcnt(1)\n\tv_mov_b32 %0, v164\n\tv_mov_b32 %1, v165\n\tv_mov_b32 %2, v166\n\tv_mov_b32 %3, v167"
                     : "=&v"(h.x), "=&v"(h.y), "=&v"(h.z), "=&v"(h.w) : "v"(vo), "s"(sb) : "memory", "v160", "v161", "v162", "v163");
        if (__all(h.x >= c.call && h.z >= c.call)) return true;
        if (give_up(c, t0, polls)) return false;
    }
}
__device__ __forceinline__ void gemm_role(const Params& P, Ctx& c, LAS char* lds, int g, int w, int lane) {
    char* myws = P.ws[P.rank];
    const int xcd = g & 7;
    STAMP(0);
    {
        int rb, cb0, ntile;
        if (g >= NRED) {
            const int i = (g - NRED) >> 3;
            rb = 8 + (xcd & 3) * 2 + (i & 1); cb0 = 8 + (xcd >> 2) * 4 + (i >> 1); ntile = 1;
        } else {
            const int i = g >> 3;
            rb = (i & 2) * 4 + (xcd & 3) * 2 + (i & 1);
            cb0 = (xcd >> 2) * 4 + (i >> 2); ntile = rb < 8 ? 2 : 1;
        }
        if (w >= 8) { gemm_loader(P, c, lds, rb, cb0, ntile, w - 8, lane); return; }
        if (GR_KV == 2) gemm_mfma<2>(P, c, lds, rb, cb0, ntile, w, lane);
        else if (GR_KV == 0 || w < 4) gemm_mfma<0>(P, c, lds, rb, cb0, ntile, w, lane);
        else gemm_mfma<GR_KV == 1 ? 1 : 3>(P, c, lds, rb, cb0, ntile, w, lane);
        if (g >= NRED) return;
    }
#define GR_REDMAP 1
#define GR_SPEC 0
    const bool late = GR_SPEC || (GR_REDMAP && (g & 16) == 0);
    const int quad = w >> 1, cidx = GR_REDMAP ? ((g & 16) << 3) + ((g & 15) | ((g >> 5) << 4)) * 2 + (w & 1) : (w & 1) * 128 + g;
    const int qm = quad >> 1, qn = quad & 1;
    const int s0 = (qm << 2) | (qn << 1);
#define GR_CLRPOS 1
    float y[32];
    bool used = false;
    {
        u32 a0 = 0, b0 = 0, a1 = 0, b1 = 0; bool ok = false, hdr = false;
        const char* hb = myws + OFF_HP + ((size_t)s0 * NPOS_OUT + cidx) * 16;
        const u32 vo = (u32)(lane >> 5) * (NPOS_OUT * 16u);
        const char* pb = myws + OFF_P + ((size_t)s0 * NPOS_OUT + cidx) * 4096;
        const u32 v0 = (u32)lane * 16u, v1 = v0 + NPOS_OUT * 4096u;
        u32x4 p0, p1, p2, p3 = {0u, 0u, 0u, 0u}, q0, q1, q2, q3 = {0u, 0u, 0u, 0u};
        if (!c.dead) {
            const u64 t0 = rt(); u32 polls = 0;
            for (;;) {
                u32x4 h;
                if (!late && !hdr) { if (!poll_all(c, hb, vo, h)) break; }
                bool fresh = false;
                if (!late && !hdr) {
                    a0 = (u32)__builtin_amdgcn_readlane((int)h.y, 0); b0 = (u32)__builtin_amdgcn_readlane((int)h.w, 0);
                    a1 = (u32)__builtin_amdgcn_readlane((int)h.y, 32); b1 = (u32)__builtin_amdgcn_readlane((int)h.w, 32);
                    hdr = true;
                    STAMP(6);
                }
                asm volatile("s_nop 4\n\tglobal_load_dwordx4 %0, %7, %9 sc0 sc1\n\tglobal_load_dwordx4 %1, %7, %9 offset:1024 sc0 sc1\n\t"
                             "global_load_dwordx4 %2, %7, %9 offset:2048 sc0 sc1\n\tglobal_load_dwordx4 %3, %8, %9 sc0 sc1\n\t"
                             "global_load_dwordx4 %4, %8, %9 offset:1024 sc0 sc1\n\tglobal_load_dwordx4 %5, %8, %9 offset:2048 sc0 sc1\n\t"
                             "global_load_dwordx4 %6, %10, %11 sc0 sc1\n\ts_waitcnt vmcnt(0)"
                             : "=&v"(p0), "=&v"(p1), "=&v"(p2), "=&v"(q0), "=&v"(q1), "=&v"(q2), "=&v"(h)
                             : "v"(v0), "v"(v1), "s"(uni(pb)), "v"(vo), "s"(uni(hb)) : "memory");
                if (!hdr && __all(h.x >= c.call && h.z >= c.call)) {
                    a0 = (u32)__builtin_amdgcn_readlane((int)h.y, 0); b0 = (u32)__builtin_amdgcn_readlane((int)h.w, 0);
                    a1 = (u32)__builtin_amdgcn_readlane((int)h.y, 32); b1 = (u32)__builtin_amdgcn_readlane((int)h.w, 32);
                    hdr = true; fresh = true;
                }
                if (hdr) {
                    if ((a0 | a1) & 0x80u) {
                        if (fresh) continue;
                        asm volatile("s_nop 4\n\tglobal_load_dwordx4 %0, %2, %4 offset:3072 sc0 sc1\n\tglobal_load_dwordx4 %1, %3, %4 offset:3072 sc0 sc1\n\ts_waitcnt vmcnt(0)"
                                     : "=&v"(p3), "=&v"(q3) : "v"(v0), "v"(v1), "s"(uni(pb)) : "memory");
                    }
                    if (((a0 & 0x80u) || arrived(p0, p1, p2)) && ((a1 & 0x80u) || arrived(q0, q1, q2))) { ok = true; break; }
                }
                if (give_up(c, t0, polls)) break;
            }
        }
        u32 d0[16], d1[16];
        if (ok) {
            used = true;
            STAMP(7);
            unpack_regs(p0, p1, p2, p3, a0, b0, lane, d0);
            unpack_regs(q0, q1, q2, q3, a1, b1, lane, d1);
        } else {
#pragma unroll
            for (int k = 0; k < 16; ++k) { d0[k] = 0; d1[k] = 0; }
        }
#pragma unroll
        for (int k = 0; k < 16; ++k) { y[2 * k] = bf_lo(d0[k]) + bf_lo(d1[k]); y[2 * k + 1] = bf_hi(d0[k]) + bf_hi(d1[k]); }
    }
    STAMP(3);
    u32 mx = 0;
#pragma unroll
    for (int k = 0; k < 32; ++k) mx = max(mx, __float_as_uint(y[k]) & 0x7FFFFFFFu);
    mx = wave_umax(mx);
    LAS u32* red = (LAS u32*)lds;
    if (lane == 0) red[w] = mx;
    __syncthreads();
    if (w == 0 && lane < NR * AMAXR) {
        u32 m = 0;
#pragma unroll
        for (int k = 0; k < 8; ++k) m = max(m, red[k]);
        if (!c.dead)
            __hip_atomic_store((GAS u64*)(P.ws[lane / AMAXR] + OFF_AMAX + (size_t)(lane % AMAXR) * 8192 + ((size_t)P.rank * NRED + g) * 8), ((u64)m << 32) | c.call, __ATOMIC_RELAXED, __HIP_MEMORY_SCOPE_SYSTEM);
    }
    if (used) {
        const char* pb = myws + OFF_P + ((size_t)s0 * NPOS_OUT + cidx) * 4096;
        clear_slot(pb, (u32)lane * 16u); clear_slot(pb, (u32)lane * 16u + NPOS_OUT * 4096u);
    }
#define GR_TMEARLY 1
    u32 tm = __hip_atomic_load((const GAS u32*)c.tmo, __ATOMIC_RELAXED, __HIP_MEMORY_SCOPE_AGENT);
    asm volatile("" : "+v"(tm));
    u32 gm = 0;
    if (!c.dead) {
        u32x4 h;
        if (poll_all(c, myws + OFF_AMAX + (size_t)(g % AMAXR) * 8192, (u32)(w * 64 + lane) * 16u, h)) gm = max(h.y, h.w);
    }
    gm = wave_umax(gm);
    if (c.dead) gm = 0xFFFFFFFFu;
    if (lane == 0) red[8 + w] = gm;
    __syncthreads();
    u32 am = 0;
#pragma unroll
    for (int k = 0; k < 8; ++k) am = max(am, red[8 + k]);
    STAMP(4);
    const bool bad = am == 0xFFFFFFFFu || tm == c.call;
    const float scale = __uint_as_float(am) / 448.0f;
    const float inv = 1.0f / scale;
    const int ph = cidx >> 7, rbh = (cidx >> 4) & 7, cb7 = (cidx >> 1) & 7, wn = cidx & 1;
    char* orow = P.out + (size_t)(qm * 256 + rbh * 32 + (lane & 15)) * 4096 + qn * 2048 + (ph * 8 + cb7) * 128 + wn * 64 + (lane >> 4) * 16;
#pragma unroll
    for (int mi = 0; mi < 2; ++mi) {
        u32x4 o;
#pragma unroll
        for (int k = 0; k < 4; ++k) {
            const float* v = y + mi * 16 + k * 4;
            int p = __builtin_amdgcn_cvt_pk_fp8_f32(v[0] * inv, v[1] * inv, 0, false);
            p = __builtin_amdgcn_cvt_pk_fp8_f32(v[2] * inv, v[3] * inv, p, true);
            o[k] = bad ? 0x7F7F7F7Fu : (u32)p;
        }
        auto s0 = __builtin_amdgcn_permlane32_swap(o[0], o[2], false, false);
        auto s1 = __builtin_amdgcn_permlane32_swap(o[1], o[3], false, false);
        auto t0 = __builtin_amdgcn_permlane16_swap(s0[0], s1[0], false, false);
        auto t1 = __builtin_amdgcn_permlane16_swap(s0[1], s1[1], false, false);
#define GR_OUTBITS "sc0 sc1"
        { const u32x4 ov = {t0[0], t0[1], t1[0], t1[1]}; const char* op = orow + (size_t)mi * 16 * 4096;
          asm volatile("global_store_dwordx4 %0, %1, off " GR_OUTBITS "\n\ts_nop 1" :: "v"(op), "v"(ov) : "memory"); }
    }
    if (g == 0 && w == 0 && lane == 0) __hip_atomic_store((GAS u32*)(myws + OFF_CTL), c.call, __ATOMIC_RELAXED, __HIP_MEMORY_SCOPE_AGENT);
    STAMP(5);
}

__global__ void __launch_bounds__(NTHR) gemm_rs_kernel(Params P) {
    extern __shared__ __attribute__((aligned(16))) char smem[];
    LAS char* lds = (LAS char*)smem;
    const int tid = threadIdx.x, w = (int)rfl((u32)tid >> 6), lane = tid & 63;
    char* myws = P.ws[P.rank];
    Ctx c;
    c.tmo = myws + OFF_CTL + 128;
    asm volatile("global_load_dword v163, %0, off sc1" :: "v"(myws + OFF_CTL) : "memory", "v163");
    const bool early = bid() >= NGEMM && bid() < NGEMM + NPUSHWG && w < PW;
    c.call = 0;
    if (!early) { WAITV(0); c.call = call_number(); }
    c.dead = false;
    const int b = bid();
    if (b < NGEMM) { gemm_role(P, c, lds, b, w, lane); return; }
    const int cw = b - NGEMM;
    if (tid < 512) STAMP(0);
    if (cw < NPUSHWG) {
        LAS u32x4* slots = (LAS u32x4*)lds;
        if (w < PW) pusher(P, c, cw, w, lane, slots);
        else {
            BAR();
            if (w == PW) scout(P, c, cw, lane, slots);
        }
    } else {
        LAS u32* cnt = (LAS u32*)(lds + 32768 + (w >> 2) * 512);
        if ((tid & 255) < 128) cnt[tid & 127] = 0;
        __syncthreads();
        unpacker(P, c, (cw - NPUSHWG) * 3 + (w >> 2), w & 3, lane, cnt, cnt + 32, rfl((u32)(size_t)lds + 40960u + (u32)w * 8192u));
    }
}

}

extern "C" void kernel_launch(int rank, int nranks, ncclComm_t comm, hipStream_t stream,
                              void* const* d_in, const int* in_sizes, int n_in,
                              void* d_out, int out_size,
                              const int* in_dtypes, int out_dtype,
                              void* const* d_out_all,
                              void* const* d_ws_all, size_t ws_bytes) {
    (void)comm; (void)in_dtypes; (void)out_dtype; (void)d_out_all;
    if (nranks != gr::NR || (unsigned)rank >= (unsigned)gr::NR || n_in < 2 || ws_bytes < gr::WS_NEED ||
        in_sizes[0] != 4096 * 512 || in_sizes[1] != 512 * 4096 || out_size != 512 * 4096) {
        fprintf(stderr, "kernel_launch: unexpected shapes\n");
        return;
    }
    static bool attr_set = false;
    if (!attr_set) {
        hipError_t e = hipFuncSetAttribute((const void*)gr::gemm_rs_kernel, hipFuncAttributeMaxDynamicSharedMemorySize, gr::LDS_BYTES);
        if (e != hipSuccess) { fprintf(stderr, "hipFuncSetAttribute: %s\n", hipGetErrorString(e)); return; }
        attr_set = true;
    }
    gr::Params p;
    p.x = (const char*)d_in[0]; p.w = (const char*)d_in[1]; p.out = (char*)d_out;
    for (int i = 0; i < gr::NR; ++i) p.ws[i] = (char*)d_ws_all[i];
    p.rank = rank; p.pad = 0;
    gr::gemm_rs_kernel<<<dim3(256), dim3(gr::NTHR), gr::LDS_BYTES, stream>>>(p);
    hipError_t e = hipGetLastError();
    if (e != hipSuccess) fprintf(stderr, "launch: %s\n", hipGetErrorString(e));
}
```
